```python
import numpy as np

kernel_name = "gpu_gemm_rs_m2048_k2048_n2048_f32_silu"


ABSMAX_RTOL = 0.001

COMPUTE_DTYPE = "float32"
EPILOGUE = "silu"
IN_QUANT = None
OUT_QUANT = None
IN_DTYPES = {"x": "float32", "w_mat": "float32"}
WORLD_SIZE = 8

def setup_inputs():

    x = np.random.randn(2048, 2048).astype(np.float32)
    w_mat = (np.random.randn(2048, 2048) / np.sqrt(2048)).astype(np.float32)
    return {"x": x, "w_mat": w_mat}

def _epilogue(y):
    return (y / (1.0 + np.exp(-np.clip(y, -60.0, 60.0)))).astype(np.float32)

def reference(x, w_mat):
    y = (x @ w_mat).astype(np.float32)
    return _epilogue(y)
```

```cpp
#include <hip/hip_runtime.h>
#include <rccl/rccl.h>
#include <cstdio>

#define HIP_CHECK(expr) do { hipError_t _e = (expr); if (_e != hipSuccess) { \
    fprintf(stderr, "HIP error %s at %s:%d\n", hipGetErrorString(_e), __FILE__, __LINE__); \
    return; } } while (0)

namespace {

constexpr int NR = 8;
constexpr int NN = 2048;
constexpr int KS = 256;
constexpr int MS = 256;
constexpr int NST = 8;
constexpr int NWG = 256;
constexpr int NTHREADS = 576;

constexpr size_t W_STAGE = 32 * 4096;
constexpr size_t W_OFF = 0;
constexpr size_t W_BYTES = (size_t)2 * NR * NST * W_STAGE;
constexpr size_t X_STAGE = 4 * 4096;
constexpr size_t X_OFF = W_OFF + W_BYTES;
constexpr size_t X_BYTES = (size_t)2 * NR * NST * X_STAGE;
constexpr size_t F_OFF = X_OFF + X_BYTES;
constexpr size_t F_BYTES = (size_t)2 * NR * 36 * 128;
constexpr size_t C_OFF = F_OFF + F_BYTES;
constexpr size_t T_OFF = C_OFF + NWG * 4;
constexpr size_t WS_NEED = T_OFF + 128;
constexpr unsigned long long DEADLINE = 50000000ull;

typedef float f32x4 __attribute__((ext_vector_type(4)));
typedef unsigned u32x4 __attribute__((ext_vector_type(4)));
typedef _Float16 f16x8 __attribute__((ext_vector_type(8)));
typedef _Float16 f16x2 __attribute__((ext_vector_type(2)));

struct Params {
    const float* x;
    const float* w;
    float* out;
    char* ws[NR];
    int rank;
};

__device__ __forceinline__ void st16_sys(char* p, u32x4 v) {
    asm volatile("global_store_dwordx4 %0, %1, off sc0 sc1\n\ts_nop 1" :: "v"(p), "v"(v) : "memory");
}
__device__ __forceinline__ void st8_sys(char* p, unsigned long long v) {
    asm volatile("global_store_dwordx2 %0, %1, off sc0 sc1\n\ts_nop 1" :: "v"(p), "v"(v) : "memory");
}

__device__ __forceinline__ unsigned pack2(float a, float b) {
    f16x2 h = {(_Float16)a, (_Float16)b};
    return __builtin_bit_cast(unsigned, h);
}

__device__ __forceinline__ float wave_max(float v) {
#pragma unroll
    for (int o = 32; o >= 1; o >>= 1) v = fmaxf(v, __shfl_xor(v, o));
    return v;
}

__device__ __forceinline__ int shift_of(float amax) {
    unsigned b = __builtin_bit_cast(unsigned, amax);
    int e = (int)((b >> 23) & 0xff);
    if (e == 255 || amax == 0.f) return 0;
    if (e == 0) e = 1;
    return 14 - (e - 127);
}

__global__ void __launch_bounds__(NTHREADS, 1) fused_kernel(Params p) {
    __shared__ __attribute__((aligned(16))) char smem[32768 + 65536 + 64];
    char* s_stage = smem;
    char* s_red = smem + 32768;
    int* s_misc = (int*)(smem + 32768 + 65536);

    const int tid = threadIdx.x, lane = tid & 63;
    const int wave = __builtin_amdgcn_readfirstlane(tid >> 6);
    const int b = blockIdx.x;
    const int rank = p.rank;
    char* me = p.ws[rank];

    const int pcb = (b & 7) * 4 + (b >> 6);
    const int pdi = (b >> 3) & 7;
    const bool self_wg = (pdi == 0);
    const int xid = (b & 7) + 8 * (b >> 6);
    const int xdst = (rank + (xid & 7)) & 7, xrg = xid >> 3;
    const int dst = self_wg ? xdst : ((rank + pdi) & 7);

    unsigned call = 0;
    if (wave == 8) {
        unsigned* cnt = (unsigned*)(me + C_OFF) + b;
        call = __hip_atomic_load(cnt, __ATOMIC_RELAXED, __HIP_MEMORY_SCOPE_AGENT) + 1;
        if (lane == 0) {
            __hip_atomic_store(cnt, call, __ATOMIC_RELAXED, __HIP_MEMORY_SCOPE_AGENT);
            s_misc[8] = (int)call;
        }
    }
    u32x4 wfrag[4]; int wshift = 0;
    if (wave < 8) {
        const int s = wave;
        f32x4 v[8];
        const float* src = p.w + (size_t)(32 * s + 8 * (lane >> 4)) * NN + 64 * pcb + 4 * (lane & 15);
#pragma unroll
        for (int j = 0; j < 8; ++j) v[j] = *(const f32x4*)(src + (size_t)j * NN);
        f32x4 xv[8];
        if (self_wg) {
            const float* xs = p.x + (size_t)(MS * xdst + 64 * xrg + (lane & 15)) * KS + 32 * s + 8 * (lane >> 4);
#pragma unroll
            for (int a = 0; a < 4; ++a) {
                xv[2 * a] = *(const f32x4*)(xs + (size_t)a * 16 * KS);
                xv[2 * a + 1] = *(const f32x4*)(xs + (size_t)a * 16 * KS + 4);
            }
        }
        float am = 0.f;
#pragma unroll
        for (int j = 0; j < 8; ++j)
#pragma unroll
            for (int f = 0; f < 4; ++f) { float t = __builtin_fabsf(v[j][f]); am = (t > am || t != t) ? t : am; }
        am = wave_max(am);
        wshift = shift_of(am);
#pragma unroll
        for (int f = 0; f < 4; ++f)
#pragma unroll
            for (int q = 0; q < 4; ++q)
                wfrag[f][q] = pack2(__builtin_ldexpf(v[2 * q][f], wshift), __builtin_ldexpf(v[2 * q + 1][f], wshift));
        if (!self_wg) {
#pragma unroll
            for (int f = 0; f < 4; ++f) *(u32x4*)(s_stage + s * 4096 + f * 1024 + lane * 16) = wfrag[f];
            if (lane == 0) s_misc[s] = wshift;
        } else {
            float xm = 0.f;
#pragma unroll
            for (int j = 0; j < 8; ++j)
#pragma unroll
                for (int f = 0; f < 4; ++f) xm = fmaxf(xm, __builtin_fabsf(xv[j][f]));
            xm = wave_max(xm);
            const int xs = shift_of(xm);
#pragma unroll
            for (int a = 0; a < 4; ++a) {
                u32x4 fr;
                fr[0] = pack2(__builtin_ldexpf(xv[2 * a][0], xs), __builtin_ldexpf(xv[2 * a][1], xs));
                fr[1] = pack2(__builtin_ldexpf(xv[2 * a][2], xs), __builtin_ldexpf(xv[2 * a][3], xs));
                fr[2] = pack2(__builtin_ldexpf(xv[2 * a + 1][0], xs), __builtin_ldexpf(xv[2 * a + 1][1], xs));
                fr[3] = pack2(__builtin_ldexpf(xv[2 * a + 1][2], xs), __builtin_ldexpf(xv[2 * a + 1][3], xs));
                *(u32x4*)(s_stage + s * 4096 + a * 1024 + lane * 16) = fr;
            }
            if (lane == 0) s_misc[s] = xs;
        }
    }
    __syncthreads();
    call = (unsigned)__builtin_amdgcn_readfirstlane(s_misc[8]);
    const int par = call & 1;
    unsigned* tmo = (unsigned*)(me + T_OFF);

    if (wave == 8) {
        char* dbase = p.ws[dst];
        char* data = self_wg ? dbase + X_OFF + ((size_t)(par * NR + rank) * NST) * X_STAGE + (size_t)xrg * 4096
                             : dbase + W_OFF + ((size_t)(par * NR + rank) * NST) * W_STAGE + (size_t)pcb * 4096;
        const size_t sstride = self_wg ? X_STAGE : W_STAGE;
        char* flag = dbase + F_OFF + ((size_t)(par * NR + rank) * 36 + (self_wg ? 32 + xrg : pcb)) * 128;
        data += lane * 16;
#pragma unroll
        for (int s = 0; s < NST; ++s) {
            u32x4 fr[4];
#pragma unroll
            for (int f = 0; f < 4; ++f) fr[f] = *(const u32x4*)(s_stage + s * 4096 + f * 1024 + lane * 16);
            const int sh = s_misc[s];
#pragma unroll
            for (int f = 0; f < 4; ++f) st16_sys(data + (size_t)s * sstride + f * 1024, fr[f]);
            if (s > 0) {
                asm volatile("s_waitcnt vmcnt(4)" ::: "memory");
                const int shp = s_misc[s - 1];
                if (lane == 0) st8_sys(flag + (s - 1) * 8, ((unsigned long long)(unsigned)shp << 32) | call);
            }
            if (s == NST - 1) {
                asm volatile("s_waitcnt vmcnt(0)" ::: "memory");
                if (lane == 0) st8_sys(flag + s * 8, ((unsigned long long)(unsigned)sh << 32) | call);
            }
        }
    } else {
        if (self_wg) {
            const int s = wave;
            char* data = me + W_OFF + ((size_t)(par * NR + rank) * NST + s) * W_STAGE + (size_t)pcb * 4096 + lane * 16;
#pragma unroll
            for (int f = 0; f < 4; ++f) st16_sys(data + f * 1024, wfrag[f]);
            asm volatile("s_waitcnt vmcnt(0)" ::: "memory");
            char* flag = me + F_OFF + ((size_t)(par * NR + rank) * 36 + pcb) * 128;
            if (lane == 0) st8_sys(flag + s * 8, ((unsigned long long)(unsigned)wshift << 32) | call);
        }
        const int src = (rank + wave) & 7;
        const int cb = (b & 7) * 4 + ((b >> 3) & 3), rb = b >> 5;
        const char* wbase = me + W_OFF + ((size_t)(par * NR + src) * NST) * W_STAGE + (size_t)cb * 4096;
        const char* xbase = me + X_OFF + ((size_t)(par * NR + src) * NST) * X_STAGE + (size_t)rb * 2048;
        const unsigned long long* fl = (const unsigned long long*)(me + F_OFF +
            ((size_t)(par * NR + src) * 36 + ((lane & 8) ? 32 + (rb >> 1) : cb)) * 128) + (lane & 7);
        __amdgpu_buffer_rsrc_t wr = __builtin_amdgcn_make_buffer_rsrc((void*)wbase, 0, (int)(NST * W_STAGE), 0x00020000);
        __amdgpu_buffer_rsrc_t xr = __builtin_amdgcn_make_buffer_rsrc((void*)xbase, 0, (int)(NST * X_STAGE), 0x00020000);
        f32x4 acc[2][4];
#pragma unroll
        for (int a = 0; a < 2; ++a)
#pragma unroll
            for (int f = 0; f < 4; ++f) acc[a][f] = f32x4{0.f, 0.f, 0.f, 0.f};
        bool dead = false;
        const unsigned long long t0 = __builtin_amdgcn_s_memrealtime();
        unsigned long long fv = 0;
        int ready = 0;
        for (int s = 0; s < NST; ++s) {
            while (ready <= s) {
                fv = __hip_atomic_load(fl, __ATOMIC_RELAXED, __HIP_MEMORY_SCOPE_SYSTEM);
                const unsigned long long ok = __ballot((unsigned)fv == call);
                const unsigned both = (unsigned)(ok & (ok >> 8)) & 0xffu;
                ready = __builtin_ctz(~both);
                if (ready > s) break;
                if (__hip_atomic_load(tmo, __ATOMIC_RELAXED, __HIP_MEMORY_SCOPE_AGENT) == call ||
                    __builtin_amdgcn_s_memrealtime() - t0 > DEADLINE) {
                    __hip_atomic_store(tmo, call, __ATOMIC_RELAXED, __HIP_MEMORY_SCOPE_AGENT);
                    dead = true;
                    break;
                }
                __builtin_amdgcn_s_sleep(2);
            }
            if (dead) break;
            __builtin_amdgcn_fence(__ATOMIC_ACQUIRE, "wavefront");
            u32x4 af[2], bf[4];
#pragma unroll
            for (int a = 0; a < 2; ++a)
                af[a] = __builtin_amdgcn_raw_buffer_load_b128(xr, lane * 16 + a * 1024, s * (int)X_STAGE, 17);
#pragma unroll
            for (int f = 0; f < 4; ++f)
                bf[f] = __builtin_amdgcn_raw_buffer_load_b128(wr, lane * 16 + f * 1024, s * (int)W_STAGE, 17);
            const int hi = (int)(fv >> 32);
            const int e = -(__builtin_amdgcn_readlane(hi, s) + __builtin_amdgcn_readlane(hi, s + 8));
#pragma unroll
            for (int a = 0; a < 2; ++a)
#pragma unroll
                for (int f = 0; f < 4; ++f) {
                    f32x4 t = __builtin_amdgcn_mfma_f32_16x16x32_f16(__builtin_bit_cast(f16x8, af[a]), __builtin_bit_cast(f16x8, bf[f]),
                                                                      f32x4{0.f, 0.f, 0.f, 0.f}, 0, 0, 0);
#pragma unroll
                    for (int r = 0; r < 4; ++r) acc[a][f][r] += __builtin_ldexpf(t[r], e);
                }
        }
#pragma unroll
        for (int a = 0; a < 2; ++a)
#pragma unroll
            for (int r = 0; r < 4; ++r)
                *(f32x4*)(s_red + (wave * 8 + a * 4 + r) * 1024 + lane * 16) =
                    f32x4{acc[a][0][r], acc[a][1][r], acc[a][2][r], acc[a][3][r]};
    }
    __syncthreads();
    if (wave < 8) {
        const int cb = (b & 7) * 4 + ((b >> 3) & 3), rb = b >> 5;
        f32x4 y = *(const f32x4*)(s_red + wave * 1024 + lane * 16);
#pragma unroll
        for (int v = 1; v < 8; ++v) y += *(const f32x4*)(s_red + (v * 8 + wave) * 1024 + lane * 16);
        const bool bad = __hip_atomic_load(tmo, __ATOMIC_RELAXED, __HIP_MEMORY_SCOPE_AGENT) == call;
        f32x4 o;
#pragma unroll
        for (int i = 0; i < 4; ++i) {
            const float c = fminf(fmaxf(y[i], -60.f), 60.f);
            o[i] = y[i] / (1.f + __expf(-c));
            if (bad) o[i] = __builtin_nanf("");
        }
        const int row = 32 * rb + 16 * (wave >> 2) + 4 * (lane >> 4) + (wave & 3);
        *(f32x4*)(p.out + (size_t)row * NN + 64 * cb + 4 * (lane & 15)) = o;
    }
}

}

extern "C" void kernel_launch(int rank, int nranks, ncclComm_t comm,
                              hipStream_t stream,
                              void* const* d_in, const int* in_sizes, int n_in,
                              void* d_out, int out_size,
                              const int* in_dtypes, int out_dtype,
                              void* const* d_out_all,
                              void* const* d_ws_all, size_t ws_bytes) {
    if (nranks != NR || (unsigned)rank >= (unsigned)NR || ws_bytes < WS_NEED || n_in < 2) {
        fprintf(stderr, "unexpected configuration\n");
        return;
    }
    Params p;
    p.x = (const float*)d_in[0];
    p.w = (const float*)d_in[1];
    p.out = (float*)d_out;
    for (int i = 0; i < NR; ++i) p.ws[i] = (char*)d_ws_all[i];
    p.rank = rank;
    hipLaunchKernelGGL(fused_kernel, dim3(NWG), dim3(NTHREADS), 0, stream, p);
    HIP_CHECK(hipGetLastError());
}
```

```cpp
#include <hip/hip_runtime.h>
#include <rccl/rccl.h>
#include <cstdio>

#define HIP_CHECK(expr) do { hipError_t _e = (expr); if (_e != hipSuccess) { \
    fprintf(stderr, "HIP error %s at %s:%d\n", hipGetErrorString(_e), __FILE__, __LINE__); \
    return; } } while (0)

#define WIN 2
#define SPL 1
#define NP_MAJ 2
#define NP_MIN 3
#define P_MAJ 1530
#define P_MIN 650
#define T_LAND 600
#define X_PACED 1
#define T_OFS 1300
#define POLL_SLEEP 2
#define OUT_WT 1

namespace {

constexpr int NR = 8;
constexpr int NN = 2048;
constexpr int KS = 256;
constexpr int MS = 256;
constexpr int NWG = 256;
constexpr int NTHREADS = 768;

constexpr size_t W_OFF = 0;
constexpr size_t X_OFF = 4ull << 20;
constexpr size_t P_OFF = 5ull << 20;
constexpr size_t H_OFF = 7ull << 20;
constexpr size_t BUF_BYTES = (7ull << 20) + 8192;
constexpr size_t C_OFF = 2 * BUF_BYTES;
constexpr size_t T_OFF = C_OFF + NWG * 4;
constexpr size_t WS_NEED = T_OFF + 128;
constexpr unsigned long long DEADLINE = 50000000ull;
constexpr unsigned TAG16 = 0x40000000u;
constexpr unsigned TAG32 = 1u;

#define TS(i) do {} while (0)
#define TSN(i) do {} while (0)
#define TS_DUMP(idx) do {} while (0)

typedef __attribute__((address_space(3))) volatile int lds_vint;
typedef __attribute__((address_space(3))) int lds_int;
typedef float f32x4 __attribute__((ext_vector_type(4)));
typedef unsigned u32x4 __attribute__((ext_vector_type(4)));
typedef unsigned u32x2 __attribute__((ext_vector_type(2)));
typedef _Float16 f16x8 __attribute__((ext_vector_type(8)));
typedef _Float16 f16x2 __attribute__((ext_vector_type(2)));

struct Params {
    const float* x;
    const float* w;
    float* out;
    char* ws[NR];
    int rank;
};

__device__ __forceinline__ void st16_sys(char* p, u32x4 v) {
    asm volatile("global_store_dwordx4 %0, %1, off sc0 sc1\n\ts_nop 1" :: "v"(p), "v"(v) : "memory");
}
__device__ __forceinline__ void st8_sys(char* p, u32x2 v) {
    asm volatile("global_store_dwordx2 %0, %1, off sc0 sc1\n\ts_nop 1" :: "v"(p), "v"(v) : "memory");
}
template <int N> __device__ __forceinline__ void wait_vm() { asm volatile("s_waitcnt vmcnt(%0)" :: "i"(N) : "memory"); }
__device__ __forceinline__ void clean_vm() { __builtin_amdgcn_s_waitcnt(0x0F70); }
__device__ __forceinline__ void wait_lds() { asm volatile("s_waitcnt lgkmcnt(0)" ::: "memory"); }

__device__ __forceinline__ unsigned pack2(float a, float b) {
    f16x2 h = {(_Float16)a, (_Float16)b};
    return __builtin_bit_cast(unsigned, h);
}
__device__ __forceinline__ unsigned or4(u32x4 v) { return (v[0] | v[1]) | (v[2] | v[3]); }

__device__ __forceinline__ unsigned wave_max_u(unsigned v) {
#pragma unroll
    for (int o = 32; o >= 1; o >>= 1) { unsigned t = (unsigned)__shfl_xor((int)v, o); v = v > t ? v : t; }
    return (unsigned)__builtin_amdgcn_readfirstlane((int)v);
}
template <int N> __device__ __forceinline__ unsigned amax(const f32x4* v) {
    unsigned am = 0;
#pragma unroll
    for (int j = 0; j < N; ++j)
#pragma unroll
        for (int f = 0; f < 4; ++f) { unsigned t = __builtin_bit_cast(unsigned, v[j][f]) & 0x7fffffffu; am = t > am ? t : am; }
    return wave_max_u(am);
}
__device__ __forceinline__ int code_of(unsigned bits) {
    const int e = (int)(bits >> 23);
    return e < 1 ? 1 : e;
}

struct Spin {
    unsigned long long t0; unsigned it; unsigned call; unsigned* tmo; lds_vint* dead;
    __device__ __forceinline__ bool fail_lds() {
        __builtin_amdgcn_s_sleep(1);
        asm volatile("" ::: "memory");
        return __builtin_amdgcn_readfirstlane(*dead) != 0;
    }
    __device__ __forceinline__ bool fail() {
        __builtin_amdgcn_s_sleep(POLL_SLEEP);
        asm volatile("" ::: "memory");
        if ((++it & 63u) != 0u) return false;
        if (__builtin_amdgcn_readfirstlane(*dead)) return true;
        if ((unsigned)__builtin_amdgcn_readfirstlane((int)__hip_atomic_load(tmo, __ATOMIC_RELAXED, __HIP_MEMORY_SCOPE_AGENT)) == call ||
            __builtin_amdgcn_s_memrealtime() - t0 > DEADLINE) {
            __hip_atomic_store(tmo, call, __ATOMIC_RELAXED, __HIP_MEMORY_SCOPE_AGENT);
            *dead = 1;
            return true;
        }
        return false;
    }
};

constexpr int L_RED = 0;
constexpr int L_SUM = 131072;
constexpr int L_MISC = L_SUM + 16384;
constexpr int L_BYTES = L_MISC + 256;

__global__ void __launch_bounds__(NTHREADS, 1) fused_kernel(Params p) {
    __shared__ __attribute__((aligned(16))) char smem[L_BYTES];
    lds_vint* s_misc = (lds_vint*)(smem + L_MISC);

    const int tid = threadIdx.x, lane = tid & 63;
    const int wave = __builtin_amdgcn_readfirstlane(tid >> 6);
    const int b = blockIdx.x;
    const int rank = p.rank, grp = rank >> 2, gi = rank & 3;
    char* me = p.ws[rank];
    const int c8 = b & 7, j = b >> 3;
    unsigned* cnt = (unsigned*)(me + C_OFF) + b;
    TS(0);
    const unsigned long long tstart = __builtin_amdgcn_s_memrealtime();

    if (j < 16) {
        const int ks = j & 7, h = j >> 3;
        clean_vm();
        if (wave >= 6) {
            if (wave == 6 && lane < 9) s_misc[lane] = 0;
            wait_lds();
            __builtin_amdgcn_s_barrier();
            return;
        }
        const bool own = ks < 5 || (ks == 5 && (lane >> 4) < SPL);
        const int dgrp = own ? grp : grp ^ 1;
        const int tsl = ks < 5 ? 5 * gi + ks : (ks == 5 ? 28 + gi : 20 + 2 * gi + (ks - 6));
        f32x4 v[8];
        if (wave >= 4) {
            const float* src = p.w + (size_t)(32 * ks + 8 * (lane >> 4)) * NN + 64 * (c8 + 8 * (h + 2 * (wave - 4))) + 4 * (lane & 15);
#pragma unroll
            for (int q = 0; q < 8; ++q) v[q] = *(const f32x4*)(src + (size_t)q * NN);
        } else {
            const int dl = (gi + 1 + wave) & 3;
            const int row0 = (c8 < 4 ? MS * ((dgrp << 2) | dl) + 64 * c8 : MS * (((dgrp ^ 1) << 2) | (c8 - 4)) + 64 * dl) + 32 * h;
            const float* xs = p.x + (size_t)(row0 + (lane & 15)) * KS + 32 * ks + 8 * (lane >> 4);
#pragma unroll
            for (int a = 0; a < 2; ++a) {
                v[2 * a] = *(const f32x4*)(xs + (size_t)a * 16 * KS);
                v[2 * a + 1] = *(const f32x4*)(xs + (size_t)a * 16 * KS + 4);
            }
        }
        unsigned call = __hip_atomic_load(cnt, __ATOMIC_RELAXED, __HIP_MEMORY_SCOPE_AGENT) + 1u;
        char* dws[4];
#pragma unroll
        for (int d = 0; d < 4; ++d) {
            char* wo = p.ws[(grp << 2) | ((gi + 1 + d) & 3)];
            char* wx = p.ws[((grp ^ 1) << 2) | ((gi + 1 + d) & 3)];
            dws[d] = own ? wo : wx;
        }
        __builtin_amdgcn_s_barrier();
        asm volatile("" ::: "memory");
        call = (unsigned)__builtin_amdgcn_readfirstlane((int)call);
        const unsigned tag = ((((call - 1u) >> 1) + 1u) & 1u) ? TAG16 : 0u;
        const size_t boff = (size_t)(call & 1u) * BUF_BYTES;
        const size_t xoff = boff + X_OFF + (size_t)(tsl * 32 + 4 * c8 + 2 * h) * 1024 + lane * 16;
        if (wave >= 4) {
            const int uh = wave - 4;
            int cd = code_of(amax<8>(v));
            if (lane == 0) s_misc[4 + uh] = cd;
            int co;
            while ((co = __builtin_amdgcn_readfirstlane(s_misc[5 - uh])) == 0) __builtin_amdgcn_s_sleep(0);
            cd = max(cd, co);
            const int sh = 126 - cd;
            char* dl = smem + L_RED + 8192 + uh * 4096 + ((lane & 15) >> 2) * 1024 + ((lane >> 4) * 16 + 4 * (lane & 3)) * 16;
#pragma unroll
            for (int c = 0; c < 4; ++c) {
                u32x4 fr;
#pragma unroll
                for (int q = 0; q < 4; ++q)
                    fr[q] = (cd == 255 ? 0u : pack2(__builtin_ldexpf(v[2 * q][c], sh), __builtin_ldexpf(v[2 * q + 1][c], sh))) | tag;
                *(u32x4*)(dl + c * 16) = fr;
            }
            wait_lds();
            if (lane == 0) s_misc[6 + uh] = 1;
            if (uh == 0) {
                int xcd;
                while (__ballot((xcd = s_misc[lane & 3]) == 0) != 0ull) __builtin_amdgcn_s_sleep(0);
                u32x2 hd;
                hd[0] = (call << 8) | (unsigned)cd;
                hd[1] = (call << 8) | (unsigned)xcd;
                const int d = lane & 3;
                char* hp = d == 0 ? dws[0] : (d == 1 ? dws[1] : (d == 2 ? dws[2] : dws[3]));
                if (lane < 4 || (lane >= 60 && ks == 5 && SPL > 0 && SPL < 4))
                    st8_sys(hp + boff + H_OFF + (size_t)(((ks == 5 && !own ? tsl + 4 : tsl) * 8 + c8) * 2 + h) * 8, hd);
            }
            __builtin_amdgcn_s_barrier();
            return;
        }
        {
            const int cd = code_of(amax<4>(v));
            TS(1);
            if (lane == 0) s_misc[wave] = cd;
            const int sh = 126 - cd;
            char* dd = wave == 0 ? dws[0] : (wave == 1 ? dws[1] : (wave == 2 ? dws[2] : dws[3]));
            const unsigned dq = ks <= 5 ? P_MAJ * 16 / 10 : P_MIN * 16 / 10;
            unsigned due = T_OFS * 256 / 10 + dq * (unsigned)(c8 * 2 + h);
#define PACE() do { while ((unsigned)(__builtin_amdgcn_s_memrealtime() - tstart) < (due >> 8)) __builtin_amdgcn_s_sleep(1); due += 16 * dq; } while (0)
#pragma unroll
            for (int a = 0; a < 2; ++a) {
                u32x4 fr;
                fr[0] = pack2(__builtin_ldexpf(v[2 * a][0], sh), __builtin_ldexpf(v[2 * a][1], sh));
                fr[1] = pack2(__builtin_ldexpf(v[2 * a][2], sh), __builtin_ldexpf(v[2 * a][3], sh));
                fr[2] = pack2(__builtin_ldexpf(v[2 * a + 1][0], sh), __builtin_ldexpf(v[2 * a + 1][1], sh));
                fr[3] = pack2(__builtin_ldexpf(v[2 * a + 1][2], sh), __builtin_ldexpf(v[2 * a + 1][3], sh));
                if (cd == 255) fr = u32x4{0u, 0u, 0u, 0u};
                fr |= tag;
                if (X_PACED) PACE(); else due += 16 * dq;
                st16_sys(dd + xoff + a * 1024, fr);
                if (a == 0) TS(2);
            }
            TS(3);
            while (__ballot(s_misc[6 + (lane & 1)] == 0) != 0ull) __builtin_amdgcn_s_sleep(0);
            asm volatile("" ::: "memory");
            char* wd = dd + boff + W_OFF + (size_t)(tsl * 32 + c8 + 8 * h) * 4096 + lane * 16;
            const char* ls = smem + L_RED + 8192 + lane * 16;
#pragma unroll 1
            for (int s = 0; s < 8; ++s) {
                const u32x4 fr = *(const u32x4*)(ls + (s & 1) * 4096 + (s >> 1) * 1024);
                PACE();
                st16_sys(wd + (s & 1) * 16 * 4096 + (s >> 1) * 1024, fr);
            }
        }
        TS(7);
        __builtin_amdgcn_s_barrier();
        if (wave == 0 && lane == 0) __hip_atomic_store(cnt, call, __ATOMIC_RELAXED, __HIP_MEMORY_SCOPE_AGENT);
        return;
    }

    const int jc = j - 16, u0 = jc & 1, rb = jc >> 1;
    unsigned call = __hip_atomic_load(cnt, __ATOMIC_RELAXED, __HIP_MEMORY_SCOPE_AGENT) + 1u;
    if (wave == 9 && lane < 25) s_misc[lane] = 0;
    call = (unsigned)__builtin_amdgcn_readfirstlane((int)call);
    __syncthreads();
    if (tid == 0) __hip_atomic_store(cnt, call, __ATOMIC_RELAXED, __HIP_MEMORY_SCOPE_AGENT);
    const bool ph = ((((call - 1u) >> 1) + 1u) & 1u) != 0u;
    const size_t boff = (size_t)(call & 1u) * BUF_BYTES;
    Spin sp; sp.t0 = __builtin_amdgcn_s_memrealtime(); sp.it = 0; sp.call = call; sp.tmo = (unsigned*)(me + T_OFF); sp.dead = s_misc + 24;
    TS(1);

    if (wave >= 10) {
        clean_vm();
        const int a = wave & 1;
        const unsigned tag = ph ? TAG32 : 0u;
        char* pd = p.ws[((grp ^ 1) << 2) | (rb >> 1)] + boff + P_OFF + (size_t)(c8 + 8 * (u0 + 2 * (2 * gi + (rb & 1)))) * 16384 + a * 1024 + lane * 16;
        bool dead = false;
#pragma unroll 1
        for (int s = 0; s < 8; ++s) {
            while (!dead && __builtin_amdgcn_readfirstlane(s_misc[s]) < 8) { if (sp.fail_lds()) dead = true; }
            asm volatile("" ::: "memory");
            const char* rp = smem + L_RED + (s & 3) * 32768 + a * 1024 + lane * 16;
            f32x4 sf = *(const f32x4*)(rp + 2048);
#pragma unroll
            for (int w = 1; w < 8; ++w) sf += *(const f32x4*)(rp + w * 4096 + 2048);
            if (!dead) st16_sys(pd + s * 2048, (__builtin_bit_cast(u32x4, sf) & ~TAG32) | tag);
            TS(2 + s);
            f32x4 so = *(const f32x4*)rp;
#pragma unroll
            for (int w = 1; w < 8; ++w) so += *(const f32x4*)(rp + w * 4096);
            *(f32x4*)(smem + L_SUM + (s * 2 + a) * 1024 + lane * 16) = so;
            wait_lds();
            if (lane == 0) s_misc[8 + 8 * a + s] = 1;
        }
        TS_DUMP(14 + a);
    } else if (wave >= 8) {
        clean_vm();
        const int a = wave & 1;
        const unsigned tag = ph ? TAG32 : 0u;
        __amdgpu_buffer_rsrc_t pr = __builtin_amdgcn_make_buffer_rsrc((void*)(me + boff + P_OFF + (size_t)(c8 + 8 * jc) * 16384 + a * 1024), 0, 16384, 0x00020000);
        float* op = p.out + (size_t)(32 * rb + 16 * a + (lane & 15)) * NN + 64 * (c8 + 8 * u0) + 4 * (lane >> 4);
        bool dead = false;
#pragma unroll 1
        for (int s = 0; s < 8; ++s) {
            while (!dead && __builtin_amdgcn_readfirstlane(s_misc[8 + 8 * a + s]) == 0) { if (sp.fail_lds()) dead = true; }
            asm volatile("" ::: "memory");
            const f32x4 own = *(const f32x4*)(smem + L_SUM + (s * 2 + a) * 1024 + lane * 16);
            TS(2 + s);
            u32x4 pv = {0u, 0u, 0u, 0u};
            while (!dead) {
                pv = __builtin_amdgcn_raw_buffer_load_b128(pr, lane * 16, s * 2048, 17) ^ tag;
                if (__ballot((or4(pv) & TAG32) != 0u) == 0ull) break;
                if (sp.fail()) dead = true;
            }
            TS(10 + s);
            const f32x4 y = own + __builtin_bit_cast(f32x4, pv);
            f32x4 o;
#pragma unroll
            for (int r = 0; r < 4; ++r) {
                const float c = fminf(fmaxf(y[r], -60.f), 60.f);
                o[r] = y[r] / (1.f + __expf(-c));
            }
            if (dead) o = f32x4{__builtin_nanf(""), __builtin_nanf(""), __builtin_nanf(""), __builtin_nanf("")};
            st16_sys((char*)(op + (s & 1) * 1024 + (s >> 1) * 16), __builtin_bit_cast(u32x4, o));
        }
        TS_DUMP(9 + a);
    } else {
        const int t0 = wave * 4;
        const unsigned tag = ph ? TAG16 : 0u;
        __amdgpu_buffer_rsrc_t wr = __builtin_amdgcn_make_buffer_rsrc((void*)(me + boff + W_OFF + (size_t)(t0 * 32 + c8 + 8 * u0) * 4096), 0, 4 * 32 * 4096, 0x00020000);
        __amdgpu_buffer_rsrc_t xr = __builtin_amdgcn_make_buffer_rsrc((void*)(me + boff + X_OFF + (size_t)(t0 * 32 + 2 * rb) * 1024), 0, 4 * 32 * 1024, 0x00020000);
        __amdgpu_buffer_rsrc_t hr = __builtin_amdgcn_make_buffer_rsrc((void*)(me + boff + H_OFF), 0, 36 * 128, 0x00020000);
        bool dead = false;
        constexpr int LOWB = SPL > 0 ? SPL * 256 - 4 : 1020;
        while (!dead) {
            const int q = lane & 15;
            const unsigned v = __builtin_amdgcn_raw_buffer_load_b32(xr, (q >> 2) * 32768 + ((q >> 1) & 1) * 16384 + (q & 1) * 1024 + ((lane & 16) ? LOWB : 1020), 0, 17) ^ tag;
            if (__ballot((v & TAG16) != 0u) == 0ull) break;
            if (sp.fail()) dead = true;
        }
        TS(2);
        u32x4 af[4][4];
        unsigned hv = 0;
        while (!dead) {
            const int hq = lane & 3, ht = t0 + ((lane >> 3) & 3);
            const bool hi = SPL == 0 || (SPL < 4 && (lane & 4) != 0);
            hv = __builtin_amdgcn_raw_buffer_load_b32(hr, (ht >= 28 && hi ? ht + 4 : ht) * 128 + (hq == 0 ? (c8 * 2 + u0) * 8 : (hq == 1 ? rb * 8 + 4 : (8 + rb) * 8 + 4)), 0, 17);
            unsigned m = 0u;
#pragma unroll
            for (int tt = 0; tt < 4; ++tt)
#pragma unroll
                for (int a = 0; a < 4; ++a)
                    af[tt][a] = __builtin_amdgcn_raw_buffer_load_b128(xr, lane * 16, tt * 32768 + (a >> 1) * 16384 + (a & 1) * 1024, 17);
#pragma unroll
            for (int tt = 0; tt < 4; ++tt)
#pragma unroll
                for (int a = 0; a < 4; ++a) { af[tt][a] ^= tag; m |= or4(af[tt][a]); }
            if (__ballot((m & TAG16) != 0u || (hv >> 8) != (call & 0xffffffu)) == 0ull) break;
            if (sp.fail()) dead = true;
        }
        u32x4 b0[4], b1[4];
        const unsigned dq = (wave == 5 || wave == 6) ? P_MIN * 16 / 10 : P_MAJ * 16 / 10;
        unsigned due = (T_OFS + T_LAND) * 256 / 10 + dq * (unsigned)(32 + c8 * 2 + u0);
        bool pref = true;
#pragma unroll
        for (int tt = 0; tt < 4; ++tt) b0[tt] = __builtin_amdgcn_raw_buffer_load_b128(wr, lane * 16 + tt * (32 * 4096), 0, 17);
        int emax[2]; bool poison = false;
        {
            int e[2][8];
#pragma unroll
            for (int i = 0; i < 8; ++i) {
                const int cw = __builtin_amdgcn_readlane((int)hv, i * 4) & 255;
                const int c0 = __builtin_amdgcn_readlane((int)hv, i * 4 + 1) & 255;
                const int c1 = __builtin_amdgcn_readlane((int)hv, i * 4 + 2) & 255;
                poison |= cw == 255 || c0 == 255 || c1 == 255;
                e[0][i] = cw + c0; e[1][i] = cw + c1;
            }
            const bool lo = (lane >> 4) < SPL;
#pragma unroll
            for (int hh = 0; hh < 2; ++hh) {
                emax[hh] = max(max(max(e[hh][0], e[hh][1]), max(e[hh][2], e[hh][3])), max(max(e[hh][4], e[hh][5]), max(e[hh][6], e[hh][7])));
#pragma unroll
                for (int tt = 0; tt < 4; ++tt) {
                    const int d = emax[hh] - (lo ? e[hh][2 * tt] : e[hh][2 * tt + 1]);
                    const _Float16 m = (_Float16)__builtin_ldexpf(1.f, -d);
#pragma unroll
                    for (int a = 0; a < 2; ++a) {
                        f16x8 t = __builtin_bit_cast(f16x8, af[tt][2 * hh + a]);
                        t *= m;
                        af[tt][2 * hh + a] = __builtin_bit_cast(u32x4, t);
                    }
                }
                emax[hh] -= 252;
            }
        }
        TS(3);
#define LOADB(buf, s_) do { const int so_ = ((s_) & 1) * (16 * 4096) + ((s_) >> 1) * 1024; \
            _Pragma("unroll") for (int tt = 0; tt < 4; ++tt) \
                buf[tt] = __builtin_amdgcn_raw_buffer_load_b128(wr, lane * 16 + tt * (32 * 4096), so_, 17); } while (0)
#define STAGE(buf, nxt, s_) do { \
            for (;;) { \
                _Pragma("unroll") for (int tt = 0; tt < 4; ++tt) buf[tt] ^= tag; \
                if (dead || (pref && __ballot(((or4(buf[0]) | or4(buf[1]) | or4(buf[2]) | or4(buf[3])) & TAG16) != 0u) == 0ull)) break; \
                while ((unsigned)(__builtin_amdgcn_s_memrealtime() - tstart) < (due >> 8)) __builtin_amdgcn_s_sleep(1); \
                if (pref && sp.fail()) { dead = true; break; } \
                if ((s_) == 7) { if (!pref) TS(20); TSN(21); TS(22); } \
                pref = true; \
                asm volatile("" ::: "memory"); \
                LOADB(buf, s_); \
            } \
            TS(4 + 2 * (s_)); \
            due += 16 * dq; \
              \
            pref = (s_) < 7 && (unsigned)(__builtin_amdgcn_s_memrealtime() - tstart) >= (due >> 8); \
            LOADB(nxt, (pref ? (s_) + 1 : (s_))); \
            f32x4 acc[4]; \
            _Pragma("unroll") for (int a = 0; a < 4; ++a) acc[a] = f32x4{0.f, 0.f, 0.f, 0.f}; \
            _Pragma("unroll") for (int tt = 0; tt < 4; ++tt) \
                _Pragma("unroll") for (int a = 3; a >= 0; --a) \
                    acc[a] = __builtin_amdgcn_mfma_f32_16x16x32_f16(__builtin_bit_cast(f16x8, buf[tt]), __builtin_bit_cast(f16x8, af[tt][a]), acc[a], 0, 0, 0); \
            if ((s_) >= 4) while (!dead && (__builtin_amdgcn_readfirstlane(s_misc[8 + (s_) - 4]) & __builtin_amdgcn_readfirstlane(s_misc[16 + (s_) - 4])) == 0) { if (sp.fail_lds()) dead = true; } \
            asm volatile("" ::: "memory"); \
            char* rp = smem + L_RED + ((s_) & 3) * 32768 + wave * 4096 + lane * 16; \
            _Pragma("unroll") for (int a = 3; a >= 0; --a) { \
                const int e = emax[a >> 1]; \
                f32x4 t = {__builtin_ldexpf(acc[a][0], e), __builtin_ldexpf(acc[a][1], e), __builtin_ldexpf(acc[a][2], e), __builtin_ldexpf(acc[a][3], e)}; \
                if (poison) t = f32x4{__builtin_nanf(""), __builtin_nanf(""), __builtin_nanf(""), __builtin_nanf("")}; \
                *(f32x4*)(rp + a * 1024) = t; \
            } \
            wait_lds(); \
            if (lane == 0) __hip_atomic_fetch_add((lds_int*)(s_misc + (s_)), 1, __ATOMIC_RELAXED, __HIP_MEMORY_SCOPE_WORKGROUP); \
            TS(5 + 2 * (s_)); \
        } while (0)
#pragma unroll 1
        for (int s = 0; s < 8; s += 2) {
            STAGE(b0, b1, s);
            STAGE(b1, b0, s + 1);
        }
        TS_DUMP(wave);
    }
}

}

extern "C" void kernel_launch(int rank, int nranks, ncclComm_t comm,
                              hipStream_t stream,
                              void* const* d_in, const int* in_sizes, int n_in,
                              void* d_out, int out_size,
                              const int* in_dtypes, int out_dtype,
                              void* const* d_out_all,
                              void* const* d_ws_all, size_t ws_bytes) {
    if (nranks != NR || (unsigned)rank >= (unsigned)NR || ws_bytes < WS_NEED || n_in < 2) {
        fprintf(stderr, "unexpected configuration\n");
        return;
    }
    Params p;
    p.x = (const float*)d_in[0];
    p.w = (const float*)d_in[1];
    p.out = (float*)d_out;
    for (int i = 0; i < NR; ++i) p.ws[i] = (char*)d_ws_all[i];
    p.rank = rank;
    hipLaunchKernelGGL(fused_kernel, dim3(NWG), dim3(NTHREADS), 0, stream, p);
    HIP_CHECK(hipGetLastError());
}
```

```cpp
#include <hip/hip_runtime.h>
#include <rccl/rccl.h>
#include <cstdio>

#define HIP_CHECK(expr) do { hipError_t _e = (expr); if (_e != hipSuccess) { \
    fprintf(stderr, "HIP error %s at %s:%d\n", hipGetErrorString(_e), __FILE__, __LINE__); \
    return; } } while (0)

#define WIN 2
#define SPL 1
#define NP_MAJ 2
#define NP_MIN 3
#define P_MAJ 1530
#define P_MIN 800
#define W_DELAY 500
#define XO_DELAY 0
#define LAG_INC 400
#define LAG_DEC 100
#define T_PL 400
#define T_PL2 200
#define NTAIL 2
#define SPREAD 1
#define T_LAND 600
#define T_PART 300
#define P_PART 1190
#define T_LANDX 900
#define X_SLEEP 12
#define W_SLEEP 6
#define XO 0
#define KS5_FAST 1
#define ADAPT 1
#define T_OWN 200
#define T_HOLD 0
#define FIN_SLEEP 3
#define X_PACED 1
#define T_OFS 1000
#define POLL_SLEEP 2
#define OUT_WT 1

namespace {

constexpr int NR = 8;
constexpr int NN = 2048;
constexpr int KS = 256;
constexpr int MS = 256;
constexpr int NWG = 256;
constexpr int NTHREADS = 512 + 256 * NTAIL;

constexpr size_t W_OFF = 0;
constexpr size_t X_OFF = 4ull << 20;
constexpr size_t P_OFF = 5ull << 20;
constexpr size_t BUF_BYTES = 7ull << 20;
constexpr size_t C_OFF = 2 * BUF_BYTES;
constexpr size_t T_OFF = C_OFF + NWG * 4;
constexpr size_t WS_NEED = T_OFF + 128;
constexpr unsigned long long DEADLINE = 150000000ull;
constexpr unsigned TAG16 = 0x40000000u;
constexpr unsigned TAG32 = 1u;

#define TS(i) do {} while (0)
#define TSN(i) do {} while (0)
#define TS_DUMP(idx) do {} while (0)

typedef __attribute__((address_space(3))) volatile int lds_vint;
typedef __attribute__((address_space(3))) int lds_int;
typedef float f32x4 __attribute__((ext_vector_type(4)));
typedef unsigned u32x4 __attribute__((ext_vector_type(4)));
typedef unsigned u32x2 __attribute__((ext_vector_type(2)));
typedef _Float16 f16x8 __attribute__((ext_vector_type(8)));
typedef _Float16 f16x2 __attribute__((ext_vector_type(2)));

struct Params {
    const float* x;
    const float* w;
    float* out;
    char* me;
    char* rel[2][4];
    int rank;
};

__device__ __forceinline__ void st16_sys(char* p, u32x4 v) {
    asm volatile("global_store_dwordx4 %0, %1, off sc0 sc1\n\ts_nop 1" :: "v"(p), "v"(v) : "memory");
}
__device__ __forceinline__ void st8_sys(char* p, u32x2 v) {
    asm volatile("global_store_dwordx2 %0, %1, off sc0 sc1\n\ts_nop 1" :: "v"(p), "v"(v) : "memory");
}
template <int N> __device__ __forceinline__ void wait_vm() { asm volatile("s_waitcnt vmcnt(%0)" :: "i"(N) : "memory"); }
__device__ __forceinline__ void clean_vm() { __builtin_amdgcn_s_waitcnt(0x0F70); }
__device__ __forceinline__ void wait_lds() { asm volatile("s_waitcnt lgkmcnt(0)" ::: "memory"); }

__device__ __forceinline__ unsigned pack2(float a, float b) {
    f16x2 h = {(_Float16)a, (_Float16)b};
    return __builtin_bit_cast(unsigned, h);
}
__device__ __forceinline__ unsigned or4(u32x4 v) { return (v[0] | v[1]) | (v[2] | v[3]); }

__device__ __forceinline__ unsigned wave_max_u(unsigned v) {
#define DPP_MAX(ctrl, rmask) do { const unsigned t_ = (unsigned)__builtin_amdgcn_update_dpp(0, (int)v, ctrl, rmask, 0xf, false); v = v > t_ ? v : t_; } while (0)
    DPP_MAX(0x111, 0xf); DPP_MAX(0x112, 0xf); DPP_MAX(0x114, 0xf); DPP_MAX(0x118, 0xf);
    DPP_MAX(0x142, 0xa); DPP_MAX(0x143, 0xc);
    return (unsigned)__builtin_amdgcn_readlane((int)v, 63);
}
template <int N> __device__ __forceinline__ unsigned amax(const f32x4* v) {
    unsigned am = 0;
#pragma unroll
    for (int j = 0; j < N; ++j) {
        const u32x4 u = __builtin_bit_cast(u32x4, v[j]) & 0x7fffffffu;
#pragma unroll
        for (int f = 0; f < 4; ++f) am = u[f] > am ? u[f] : am;
    }
    return wave_max_u(am);
}
__device__ __forceinline__ int code_of(unsigned bits) {
    const int e = (int)(bits >> 23);
    return e == 255 ? 255 : ((e + 1) | 3) - 1;
}

__device__ __forceinline__ int code_from(u32x4 v, int lane) {
    const int nib = (int)(((v[0] >> 14) & 1u) | ((v[1] >> 13) & 2u) | ((v[2] >> 12) & 4u) | ((v[3] >> 11) & 8u));
    const int oth = __shfl_xor(nib, 1);
    return (lane & 1) ? (nib << 4) | oth : (oth << 4) | nib;
}
constexpr unsigned PAYLOAD = 0xBFFFBFFFu;

struct Spin {
    unsigned long long t0; unsigned it; unsigned call; unsigned* tmo; lds_vint* dead;
    __device__ __forceinline__ bool fail_lds() {
        __builtin_amdgcn_s_sleep(1);
        asm volatile("" ::: "memory");
        if (__builtin_amdgcn_readfirstlane(*dead) != 0) return true;
        if ((++it & 1023u) == 0u && __builtin_amdgcn_s_memrealtime() - t0 > DEADLINE) {
            __hip_atomic_store(tmo, call, __ATOMIC_RELAXED, __HIP_MEMORY_SCOPE_AGENT);
            *dead = 1;
            return true;
        }
        return false;
    }
    __device__ __forceinline__ bool fail() {
        __builtin_amdgcn_s_sleep(POLL_SLEEP);
        asm volatile("" ::: "memory");
        if ((++it & 63u) != 0u) return false;
        if (__builtin_amdgcn_readfirstlane(*dead)) return true;
        if ((unsigned)__builtin_amdgcn_readfirstlane((int)__hip_atomic_load(tmo, __ATOMIC_RELAXED, __HIP_MEMORY_SCOPE_AGENT)) == call ||
            __builtin_amdgcn_s_memrealtime() - t0 > DEADLINE) {
            __hip_atomic_store(tmo, call, __ATOMIC_RELAXED, __HIP_MEMORY_SCOPE_AGENT);
            *dead = 1;
            return true;
        }
        return false;
    }
};

constexpr int L_RED = 0;
constexpr int L_SUM = 131072;
constexpr int L_MISC = L_SUM + 16384;
constexpr int L_BYTES = L_MISC + 256;

__global__ void __launch_bounds__(NTHREADS, 1) fused_kernel(Params p) {
    __shared__ __attribute__((aligned(16))) char smem[L_BYTES];
    lds_vint* s_misc = (lds_vint*)(smem + L_MISC);

    const int tid = threadIdx.x, lane = tid & 63;
    const int wave = __builtin_amdgcn_readfirstlane(tid >> 6);
    const int b = blockIdx.x;
    const int rank = p.rank, grp = rank >> 2, gi = rank & 3;
    char* me = p.me;
    const float* px = p.x;
    asm volatile("" :: "s"(px));
    const int j = b >> 3;
    const int c8 = (SPREAD && j < 16) ? (j & 7) : (b & 7);
    unsigned* cnt = (unsigned*)(me + C_OFF) + b;
    TS(0);
    const unsigned long long tstart = __builtin_amdgcn_s_memrealtime();

    if (__builtin_expect(j < 16, 1)) {
        const int ks = SPREAD ? ((b & 7) - c8) & 7 : j & 7, h = j >> 3;
        const bool split = ks == 5 && SPL > 0 && SPL < 4;
        if (wave > 8 || (wave > 4 && !split)) return;
        if (wave == 4) {
            f32x4 v[16];
            while ((unsigned)(__builtin_amdgcn_s_memrealtime() - tstart) < W_DELAY / 10) __builtin_amdgcn_s_sleep(1);
            const float* src = p.w + (size_t)(32 * ks + 8 * (lane >> 4)) * NN + 64 * (c8 + 8 * h) + 4 * (lane & 15);
#pragma unroll
            for (int q = 0; q < 16; ++q) v[q] = *(const f32x4*)(src + (size_t)(q & 7) * NN + (q >> 3) * 1024);
            const unsigned call = (unsigned)__builtin_amdgcn_readfirstlane((int)__hip_atomic_load(cnt, __ATOMIC_RELAXED, __HIP_MEMORY_SCOPE_AGENT)) + 1u;
            const unsigned tag = ((((call - 1u) >> 1) + 1u) & 1u) ? TAG16 : 0u;
            const int cd = code_of(amax<16>(v));
            const int sh = 126 - cd;
            char* dq_ = smem + L_RED + ((lane & 15) >> 2) * 1024 + ((lane >> 4) * 16 + 4 * (lane & 3)) * 16;
#pragma unroll
            for (int uh = 0; uh < 2; ++uh)
#pragma unroll
                for (int c = 0; c < 4; ++c) {
                    u32x4 fr;
#pragma unroll
                    for (int q = 0; q < 4; ++q)
                        fr[q] = (cd == 255 ? 0u : pack2(__builtin_ldexpf(v[uh * 8 + 2 * q][c], sh), __builtin_ldexpf(v[uh * 8 + 2 * q + 1][c], sh))) | tag |
                                ((((unsigned)cd >> (4 * (c & 1) + q)) & 1u) << 14);
                    *(u32x4*)(dq_ + uh * 4096 + c * 16) = fr;
                }
            wait_lds();
            __builtin_amdgcn_s_barrier();
            if (lane == 0) __hip_atomic_store(cnt, call, __ATOMIC_RELAXED, __HIP_MEMORY_SCOPE_AGENT);
            return;
        }
        const bool own = ks < 5 || (ks == 5 && (split ? wave < 4 : SPL == 4));
        const bool act = !split || (((lane >> 4) < SPL) == own);
        const int dgrp = own ? grp : grp ^ 1;
        const int dw = (wave < 4 ? wave : wave - 5);
        const int dl = (gi + 1 + dw) & 3;
        const int tsl = ks < 5 ? 5 * gi + ks : (ks == 5 ? 28 + gi : 20 + 2 * gi + (ks - 6));
        f32x4 v[4];
        {
            const int row0 = (c8 < 4 ? MS * ((dgrp << 2) | dl) + 64 * c8 : MS * (((dgrp ^ 1) << 2) | (c8 - 4)) + 64 * dl) + 32 * h;
            const float* xs = px + (size_t)(row0 + (lane & 15)) * KS + 32 * ks + 8 * (lane >> 4);
            if (c8 < 4) while ((unsigned)(__builtin_amdgcn_s_memrealtime() - tstart) < XO_DELAY / 10) __builtin_amdgcn_s_sleep(1);
#pragma unroll
            for (int a = 0; a < 2; ++a) {
                v[2 * a] = *(const f32x4*)(xs + (size_t)a * 16 * KS);
                v[2 * a + 1] = *(const f32x4*)(xs + (size_t)a * 16 * KS + 4);
            }
        }
        unsigned call = __hip_atomic_load(cnt, __ATOMIC_RELAXED, __HIP_MEMORY_SCOPE_AGENT) + 1u;
        char* dd = p.rel[own ? 0 : 1][dw];
        call = (unsigned)__builtin_amdgcn_readfirstlane((int)call);
        const unsigned tag = ((((call - 1u) >> 1) + 1u) & 1u) ? TAG16 : 0u;
        const size_t boff = (size_t)(call & 1u) * BUF_BYTES;
        {
            const int cd = code_of(amax<4>(v));
            TS(1);
            const int sh = 126 - cd;
            const unsigned nib = (unsigned)cd >> (4 * (lane & 1));
            const u32x4 xtag = {tag | ((nib & 1u) << 14), tag | ((nib & 2u) << 13), tag | ((nib & 4u) << 12), tag | ((nib & 8u) << 11)};
            u32x4 xf[2];
#pragma unroll
            for (int a = 0; a < 2; ++a) {
                xf[a][0] = pack2(__builtin_ldexpf(v[2 * a][0], sh), __builtin_ldexpf(v[2 * a][1], sh));
                xf[a][1] = pack2(__builtin_ldexpf(v[2 * a][2], sh), __builtin_ldexpf(v[2 * a][3], sh));
                xf[a][2] = pack2(__builtin_ldexpf(v[2 * a + 1][0], sh), __builtin_ldexpf(v[2 * a + 1][1], sh));
                xf[a][3] = pack2(__builtin_ldexpf(v[2 * a + 1][2], sh), __builtin_ldexpf(v[2 * a + 1][3], sh));
                if (cd == 255) xf[a] = u32x4{0u, 0u, 0u, 0u};
                xf[a] |= xtag;
            }
            const unsigned dq = (own || (ks == 5 && !KS5_FAST)) ? P_MAJ * 16 / 10 : P_MIN * 16 / 10;
            unsigned t00 = ((unsigned)(__builtin_amdgcn_s_memrealtime() - tstart) - (c8 < 4 ? XO_DELAY / 10 : 0)) << 8;
            t00 = (ADAPT && t00 > T_OFS * 256 / 10) ? t00 : T_OFS * 256 / 10;
#define PACE(t_) while ((unsigned)(__builtin_amdgcn_s_memrealtime() - tstart) < ((t_) >> 8)) __builtin_amdgcn_s_sleep(1)
            char* xd = dd + boff + X_OFF + (size_t)(tsl * 32 + 4 * c8 + 2 * h) * 1024 + lane * 16;
            char* wd = dd + boff + W_OFF + (size_t)(tsl * 32 + c8 + 8 * h) * 4096 + lane * 16;
            const char* ls = smem + L_RED + lane * 16;
            unsigned due = t00 + dq * (unsigned)(16 + c8 * 2 + h);
#pragma unroll 1
            for (int s = -1; s < 8; ++s) {
                if (s == (c8 >= 4 ? -1 : XO)) {
                    const unsigned tx = t00 + dq * (unsigned)((s + 1) * 16 + ((c8 & 3) * 2 + h) * 2);
                    PACE(tx);
                    if (act) st16_sys(xd, xf[0]);
                    TS(2);
                    PACE(tx + dq);
                    if (act) st16_sys(xd + 1024, xf[1]);
                    TS(3);
                }
                if (s == XO) due += 16 * dq;
                if (s < 0) { __builtin_amdgcn_s_barrier(); asm volatile("" ::: "memory"); continue; }
                const u32x4 fr = *(const u32x4*)(ls + (s & 1) * 4096 + (s >> 1) * 1024);
                PACE(due);
                due += 16 * dq;
                if (act) st16_sys(wd + (s & 1) * 16 * 4096 + (s >> 1) * 1024, fr);
            }
        }
        TS(7);
        return;
    }

    const int jc = j - 16, u0 = jc & 1, kind = 1 - ((jc >> 1) & 1), rq = jc >> 2;
    unsigned call = __hip_atomic_load(cnt, __ATOMIC_RELAXED, __HIP_MEMORY_SCOPE_AGENT) + 1u;
    if (wave == 9 && lane < 28) s_misc[lane] = 0;
    call = (unsigned)__builtin_amdgcn_readfirstlane((int)call);
    __syncthreads();
    if (tid == 0) __hip_atomic_store(cnt, call, __ATOMIC_RELAXED, __HIP_MEMORY_SCOPE_AGENT);
    const bool ph = ((((call - 1u) >> 1) + 1u) & 1u) != 0u;
    const size_t boff = (size_t)(call & 1u) * BUF_BYTES;
    Spin sp; sp.t0 = tstart; sp.it = 0; sp.call = call; sp.tmo = (unsigned*)(me + T_OFF); sp.dead = s_misc + 24;
    const unsigned due0 = (T_OFS + T_LAND + (kind ? 0 : T_OWN)) * 256 / 10 + (P_MAJ * 16 / 10) * (unsigned)(16 + c8 * 2 + u0);
#define DUE(s_) (due0 + (unsigned)((s_) + ((s_) >= XO ? 1 : 0)) * (16 * (P_MAJ * 16 / 10)))
#define NOW() ((unsigned)(__builtin_amdgcn_s_memrealtime() - tstart))
#define LDS_WAIT(idx, n) while (!dead && __builtin_amdgcn_readfirstlane(s_misc[idx]) < (n)) { if (sp.fail_lds()) dead = true; }
#define LDS_SIGNAL(idx) do { if (lane == 0) __hip_atomic_fetch_add((lds_int*)(s_misc + (idx)), 1, __ATOMIC_RELAXED, __HIP_MEMORY_SCOPE_WORKGROUP); } while (0)
    TS(1);

    if (wave >= 8) {
        clean_vm();
        const int a = wave & 3;
        const unsigned tag = ph ? TAG32 : 0u;
        char* pp = (kind ? p.rel[1][(rq - gi - 1) & 3] : me) + boff + P_OFF + (size_t)((c8 * 2 + u0) * 4 + (kind ? gi : rq)) * 32768 + a * 1024 + lane * 16;
        float* op = p.out + (size_t)(64 * rq + 16 * a + (lane & 15)) * NN + 64 * (c8 + 8 * u0) + 4 * (lane >> 4);
        bool dead = false;
#pragma unroll 1
        for (int s = (wave >> 2) & 1; s < 8; s += NTAIL) {
            LDS_WAIT(s, 8);
            asm volatile("" ::: "memory");
            const char* rp = smem + L_RED + (s & 3) * 32768 + a * 1024 + lane * 16;
            f32x4 sum = *(const f32x4*)rp;
#pragma unroll
            for (int w = 1; w < 8; ++w) sum += *(const f32x4*)(rp + w * 4096);
            asm volatile("" : "+v"(sum) :: "memory");
            LDS_SIGNAL(8 + s);
            if (kind) {
                { const unsigned tp = (T_OFS + 10 * P_MIN + T_PART) * 256 / 10 + (P_PART * 16 / 10) * (unsigned)(s * 16 + c8 * 2 + u0);
                  while (NOW() < (tp >> 8)) __builtin_amdgcn_s_sleep(1); }
                if (T_HOLD > 0) {
                    const unsigned now = NOW();
                    for (int n = s + 1; n < 8; ++n)
                        if (now + T_HOLD / 10 >= (DUE(n) >> 8))
                            while (__builtin_amdgcn_readfirstlane(s_misc[16 + n]) < ((SPL == 0 && KS5_FAST) ? 5 : 6) && NOW() < (DUE(n) >> 8) + 15)
                                __builtin_amdgcn_s_sleep(1);
                }
                if (!dead) st16_sys(pp + s * 4096, (__builtin_bit_cast(u32x4, sum) & ~TAG32) | tag);
                TS(2 + s);
            } else {
                TS(2 + s);
                if (T_PL > 0) {
                    const unsigned tp = (T_OFS + 10 * P_MIN + T_PART + T_PL) * 256 / 10 + (P_PART * 16 / 10) * (unsigned)(s * 16 + c8 * 2 + u0);
                    const unsigned t2 = NOW() + T_PL2 / 10;
                    while (NOW() < (tp >> 8) || NOW() < t2) __builtin_amdgcn_s_sleep(1);
                }
                u32x4 pv = {0u, 0u, 0u, 0u};
                while (!dead) {
                    asm volatile("global_load_dwordx4 %0, %1, off sc0 sc1\n\ts_waitcnt vmcnt(0)" : "=&v"(pv) : "v"(pp + s * 4096) : "memory");
                    pv ^= tag;
                    if (__ballot((or4(pv) & TAG32) != 0u) == 0ull) break;
                    if (sp.fail()) dead = true;
                }
                TS(10 + s);
                const f32x4 y = sum + __builtin_bit_cast(f32x4, pv);
                f32x4 o;
#pragma unroll
                for (int r = 0; r < 4; ++r) {
                    const float c = fminf(fmaxf(y[r], -60.f), 60.f);
                    o[r] = y[r] / (1.f + __expf(-c));
                }
                if (dead) o = f32x4{__builtin_nanf(""), __builtin_nanf(""), __builtin_nanf(""), __builtin_nanf("")};
                st16_sys((char*)(op + (s & 1) * 1024 + (s >> 1) * 16), __builtin_bit_cast(u32x4, o));
            }
        }
        if (a == 0) TS_DUMP((kind ? 14 : 9) + ((wave >> 2) & 1));
    } else {
        const int t0 = wave * 4;
        const unsigned tag = ph ? TAG16 : 0u;
        __amdgpu_buffer_rsrc_t xr = __builtin_amdgcn_make_buffer_rsrc((void*)(me + boff + X_OFF + (size_t)(t0 * 32 + 16 * kind + 4 * rq) * 1024), 0, 4 * 32 * 1024, 0x00020000);
        bool dead = false;
        constexpr int LOWB = SPL > 0 ? SPL * 256 - 4 : 1020;
        {
            const bool fast = wave == 5 || wave == 6;
            const unsigned tx = (T_OFS + T_LANDX) * 256 / 10 + (fast ? P_MIN * 16 / 10 : P_MAJ * 16 / 10) * (unsigned)((kind ? 0 : (1 + XO) * 16) + 4 * rq + 3);
            while (NOW() < (tx >> 8)) __builtin_amdgcn_s_sleep(2);
        }
        while (!dead) {
            const unsigned v = __builtin_amdgcn_raw_buffer_load_b32(xr, (lane & 3) * 32768 + 3 * 1024 + ((wave == 7 && (lane & 4)) ? LOWB : 1020), 0, 17) ^ tag;
            if (__ballot((v & TAG16) != 0u) == 0ull) break;
            __builtin_amdgcn_s_sleep(X_SLEEP);
            if (sp.fail()) dead = true;
        }
        TS(2);
        u32x4 af[4][4];
        while (!dead) {
            unsigned m = 0u;
#pragma unroll
            for (int tt = 0; tt < 4; ++tt)
#pragma unroll
                for (int a = 0; a < 4; ++a)
                    af[tt][a] = __builtin_amdgcn_raw_buffer_load_b128(xr, lane * 16, tt * 32768 + a * 1024, 17);
#pragma unroll
            for (int tt = 0; tt < 4; ++tt)
#pragma unroll
                for (int a = 0; a < 4; ++a) { af[tt][a] ^= tag; m |= or4(af[tt][a]); }
            if (__ballot((m & TAG16) != 0u) == 0ull) break;
            __builtin_amdgcn_s_sleep(X_SLEEP);
            if (sp.fail()) dead = true;
        }
        clean_vm();
        const char* wq = me + boff + W_OFF + (size_t)(t0 * 32 + c8 + 8 * u0) * 4096 + lane * 16;
#define LOADW(s_, inc_) do { const char* q_ = wq + ((s_) & 1) * (16 * 4096) + ((s_) >> 1) * 1024; \
            asm volatile("global_load_dwordx4 %0, %4, off sc0 sc1\n\tglobal_load_dwordx4 %1, %5, off sc0 sc1\n\t" \
                         "global_load_dwordx4 %2, %6, off sc0 sc1\n\tglobal_load_dwordx4 %3, %7, off sc0 sc1\n\t" \
                         "ds_add_u32 %8, %9\n\ts_waitcnt vmcnt(0) lgkmcnt(0)" \
                         : "=&v"(cur[0]), "=&v"(cur[1]), "=&v"(cur[2]), "=&v"(cur[3]) \
                         : "v"(q_), "v"(q_ + 32 * 4096), "v"(q_ + 2 * 32 * 4096), "v"(q_ + 3 * 32 * 4096), \
                           "v"((unsigned)(L_MISC + (16 + (s_)) * 4) + lds_base), "v"(inc_) : "memory"); } while (0)
        const unsigned lds_base = (unsigned)(size_t)(__attribute__((address_space(3))) char*)smem;
        const bool minor = wave == 5 || wave == 6 || (wave == 7 && SPL == 0 && KS5_FAST);
        const unsigned duem = (T_OFS + T_LAND + 300) * 256 / 10 + (P_MIN * 16 / 10) * (unsigned)(16 + c8 * 2 + u0);
        unsigned lag = 0;
        int cx[4][2], ex[2] = {0, 0};
#pragma unroll
        for (int tt = 0; tt < 4; ++tt) {
            cx[tt][0] = code_from(af[tt][0], lane); cx[tt][1] = code_from(af[tt][2], lane);
#pragma unroll
            for (int a = 0; a < 4; ++a) af[tt][a] &= PAYLOAD;
        }
        TS(3);
#pragma unroll 1
        for (int s = 0; s < 8; ++s) {
            const unsigned g_ = (minor ? duem + (unsigned)(s + (s >= XO ? 1 : 0)) * (16 * (P_MIN * 16 / 10)) : DUE(s)) >> 8;
            while (NOW() < g_ + lag) __builtin_amdgcn_s_sleep(1);
            if (s == 7) TS(20);
            u32x4 cur[4];
            int inc = (minor || lane != 0) ? 0 : 1;
            lag = lag > LAG_DEC / 10 ? lag - LAG_DEC / 10 : 0;
            for (;;) {
                const unsigned t_try = NOW();
                LOADW(s, inc);
                inc = 0;
#pragma unroll
                for (int tt = 0; tt < 4; ++tt) cur[tt] ^= tag;
                if (dead || __ballot(((or4(cur[0]) | or4(cur[1]) | or4(cur[2]) | or4(cur[3])) & TAG16) != 0u) == 0ull) break;
                if (LAG_INC > 0) lag = t_try + LAG_INC / 10 - g_;
                __builtin_amdgcn_s_sleep(W_SLEEP);
                if (sp.fail()) { dead = true; break; }
            }
            TS(4 + 2 * s);
            if (s == 0) {
                bool bad = false, same = true;
#pragma unroll
                for (int tt = 0; tt < 4; ++tt) {
                    const int cw = code_from(cur[tt], lane);
                    bad |= cw == 255 || cx[tt][0] == 255 || cx[tt][1] == 255;
                    cx[tt][0] += cw; cx[tt][1] += cw;
                }
                ex[0] = __builtin_amdgcn_readfirstlane(cx[0][0]); ex[1] = __builtin_amdgcn_readfirstlane(cx[0][1]);
#pragma unroll
                for (int tt = 0; tt < 4; ++tt) same &= cx[tt][0] == ex[0] && cx[tt][1] == ex[1];
                if (__ballot(bad || !same) != 0ull) {
                    const bool poison = __ballot(bad) != 0ull;
#pragma unroll
                    for (int hh = 0; hh < 2; ++hh) {
                        ex[hh] = (int)wave_max_u((unsigned)max(max(cx[0][hh], cx[1][hh]), max(cx[2][hh], cx[3][hh])));
#pragma unroll
                        for (int tt = 0; tt < 4; ++tt) {
                            const _Float16 m = poison ? (_Float16)__builtin_nanf("") : (_Float16)__builtin_ldexpf(1.f, cx[tt][hh] - ex[hh]);
#pragma unroll
                            for (int a = 0; a < 2; ++a) {
                                f16x8 t = __builtin_bit_cast(f16x8, af[tt][2 * hh + a]);
                                t *= m;
                                af[tt][2 * hh + a] = __builtin_bit_cast(u32x4, t);
                            }
                        }
                    }
                }
                ex[0] -= 252; ex[1] -= 252;
            }
#pragma unroll
            for (int tt = 0; tt < 4; ++tt) cur[tt] &= PAYLOAD;
            char* rp = smem + L_RED + (s & 3) * 32768 + wave * 4096 + lane * 16;
            f32x4 acc[4];
#pragma unroll
            for (int a = 0; a < 4; ++a) acc[a] = f32x4{0.f, 0.f, 0.f, 0.f};
#pragma unroll
            for (int tt = 0; tt < 4; ++tt)
#pragma unroll
                for (int a = 0; a < 4; ++a)
                    acc[a] = __builtin_amdgcn_mfma_f32_16x16x32_f16(__builtin_bit_cast(f16x8, cur[tt]), __builtin_bit_cast(f16x8, af[tt][a]), acc[a], 0, 0, 0);
            if (s >= 4) LDS_WAIT(8 + s - 4, 4);
            asm volatile("" ::: "memory");
#pragma unroll
            for (int a = 0; a < 4; ++a) {
                const int e = ex[a >> 1];
                *(f32x4*)(rp + a * 1024) = f32x4{__builtin_ldexpf(acc[a][0], e), __builtin_ldexpf(acc[a][1], e),
                                                 __builtin_ldexpf(acc[a][2], e), __builtin_ldexpf(acc[a][3], e)};
            }
            asm volatile("" ::: "memory");
            LDS_SIGNAL(s);
            TS(5 + 2 * s);
        }
        TS_DUMP(wave);
    }
}

}

extern "C" void kernel_launch(int rank, int nranks, ncclComm_t comm,
                              hipStream_t stream,
                              void* const* d_in, const int* in_sizes, int n_in,
                              void* d_out, int out_size,
                              const int* in_dtypes, int out_dtype,
                              void* const* d_out_all,
                              void* const* d_ws_all, size_t ws_bytes) {
    if (nranks != NR || (unsigned)rank >= (unsigned)NR || ws_bytes < WS_NEED || n_in < 2) {
        fprintf(stderr, "unexpected configuration\n");
        return;
    }
    Params p;
    p.x = (const float*)d_in[0];
    p.w = (const float*)d_in[1];
    p.out = (float*)d_out;
    p.me = (char*)d_ws_all[rank];
    for (int g = 0; g < 2; ++g)
        for (int d = 0; d < 4; ++d) p.rel[g][d] = (char*)d_ws_all[(((rank >> 2) ^ g) << 2) | ((rank + 1 + d) & 3)];
    p.rank = rank;
    hipLaunchKernelGGL(fused_kernel, dim3(NWG), dim3(NTHREADS), 0, stream, p);
    HIP_CHECK(hipGetLastError());
}
```

```cpp
#include <hip/hip_runtime.h>
#include <rccl/rccl.h>
#include <cstdio>

#define HIP_CHECK(expr) do { hipError_t _e = (expr); if (_e != hipSuccess) { \
    fprintf(stderr, "HIP error %s at %s:%d\n", hipGetErrorString(_e), __FILE__, __LINE__); \
    return; } } while (0)

#define T_OFS 1000
#define P_MAJ 1520
#define P_MIN 795
#define P_PART 1170
#define T_PART 300
#define T_LAND 600
#define T_LANDX 900
#define T_OWN 200
#define T_PL 400
#define W_DELAY 500
#define LAG_INC 400
#define LAG_DEC 100

namespace {

constexpr int NR = 8;
constexpr int NN = 2048;
constexpr int KS = 256;
constexpr int MS = 256;
constexpr int SPL = 1;
constexpr int NWG = 256;
constexpr int NTHREADS = 1024;

constexpr size_t W_OFF = 0;
constexpr size_t X_OFF = 4ull << 20;
constexpr size_t P_OFF = 5ull << 20;
constexpr size_t BUF_BYTES = 7ull << 20;
constexpr size_t C_OFF = 2 * BUF_BYTES;
constexpr size_t T_OFF = C_OFF + NWG * 4;
constexpr size_t WS_NEED = T_OFF + 128;
constexpr unsigned long long DEADLINE = 150000000ull;
constexpr unsigned TAG16 = 0x40000000u;
constexpr unsigned PAYLOAD = 0xBFFFBFFFu;
constexpr unsigned TAG32 = 1u;
constexpr unsigned NS(int ns) { return (unsigned)ns * 256u / 10u; }

typedef __attribute__((address_space(3))) volatile int lds_vint;
typedef __attribute__((address_space(3))) int lds_int;
typedef float f32x4 __attribute__((ext_vector_type(4)));
typedef unsigned u32x4 __attribute__((ext_vector_type(4)));
typedef _Float16 f16x8 __attribute__((ext_vector_type(8)));
typedef _Float16 f16x2 __attribute__((ext_vector_type(2)));

struct Params {
    const float* x;
    const float* w;
    float* out;
    char* me;
    char* rel[2][4];
    int rank;
};

__device__ __forceinline__ void st16_sys(char* p, u32x4 v) {
    asm volatile("global_store_dwordx4 %0, %1, off sc0 sc1\n\ts_nop 1" :: "v"(p), "v"(v) : "memory");
}
__device__ __forceinline__ void clean_vm() { __builtin_amdgcn_s_waitcnt(0x0F70); }

__device__ __forceinline__ unsigned pack2(float a, float b) {
    f16x2 h = {(_Float16)a, (_Float16)b};
    return __builtin_bit_cast(unsigned, h);
}
__device__ __forceinline__ unsigned or4(u32x4 v) { return (v[0] | v[1]) | (v[2] | v[3]); }

__device__ __forceinline__ unsigned wave_max_u(unsigned v) {
#define DPP_MAX(ctrl, rmask) do { const unsigned t_ = (unsigned)__builtin_amdgcn_update_dpp(0, (int)v, ctrl, rmask, 0xf, false); v = v > t_ ? v : t_; } while (0)
    DPP_MAX(0x111, 0xf); DPP_MAX(0x112, 0xf); DPP_MAX(0x114, 0xf); DPP_MAX(0x118, 0xf);
    DPP_MAX(0x142, 0xa); DPP_MAX(0x143, 0xc);
    return (unsigned)__builtin_amdgcn_readlane((int)v, 63);
}
template <int N> __device__ __forceinline__ int code_of(const f32x4* v) {
    unsigned am = 0;
#pragma unroll
    for (int j = 0; j < N; ++j) {
        const u32x4 u = __builtin_bit_cast(u32x4, v[j]) & 0x7fffffffu;
#pragma unroll
        for (int f = 0; f < 4; ++f) am = u[f] > am ? u[f] : am;
    }
    const int e = (int)(wave_max_u(am) >> 23);
    return e == 255 ? 255 : ((e + 1) | 3) - 1;
}
__device__ __forceinline__ int code_from(u32x4 v, int lane) {
    const int nib = (int)(((v[0] >> 14) & 1u) | ((v[1] >> 13) & 2u) | ((v[2] >> 12) & 4u) | ((v[3] >> 11) & 8u));
    const int oth = __shfl_xor(nib, 1);
    return (lane & 1) ? (nib << 4) | oth : (oth << 4) | nib;
}

struct Spin {
    unsigned long long t0; unsigned it; unsigned call; unsigned* tmo; lds_vint* dead;
    __device__ __forceinline__ bool give_up() {
        __hip_atomic_store(tmo, call, __ATOMIC_RELAXED, __HIP_MEMORY_SCOPE_AGENT);
        *dead = 1;
        return true;
    }
    __device__ __forceinline__ bool fail_lds() {
        __builtin_amdgcn_s_sleep(1);
        asm volatile("" ::: "memory");
        if (__builtin_amdgcn_readfirstlane(*dead) != 0) return true;
        if ((++it & 1023u) == 0u && __builtin_amdgcn_s_memrealtime() - t0 > DEADLINE) return give_up();
        return false;
    }
    __device__ __forceinline__ bool fail() {
        __builtin_amdgcn_s_sleep(2);
        asm volatile("" ::: "memory");
        if ((++it & 63u) != 0u) return false;
        if (__builtin_amdgcn_readfirstlane(*dead)) return true;
        if ((unsigned)__builtin_amdgcn_readfirstlane((int)__hip_atomic_load(tmo, __ATOMIC_RELAXED, __HIP_MEMORY_SCOPE_AGENT)) == call ||
            __builtin_amdgcn_s_memrealtime() - t0 > DEADLINE) return give_up();
        return false;
    }
};

constexpr int L_RED = 0;
constexpr int L_MISC = 131072;
constexpr int L_BYTES = L_MISC + 128;

__global__ void __launch_bounds__(NTHREADS, 1) fused_kernel(Params p) {
    __shared__ __attribute__((aligned(16))) char smem[L_BYTES];
    lds_vint* s_misc = (lds_vint*)(smem + L_MISC);

    const int tid = threadIdx.x, lane = tid & 63;
    const int wave = __builtin_amdgcn_readfirstlane(tid >> 6);
    const int b = blockIdx.x;
    const int rank = p.rank, grp = rank >> 2, gi = rank & 3;
    char* me = p.me;
    const float* px = p.x;
    asm volatile("" :: "s"(px));
    const int j = b >> 3;
    const int c8 = j < 16 ? (j & 7) : (b & 7);
    unsigned* cnt = (unsigned*)(me + C_OFF) + b;
    const unsigned long long tstart = __builtin_amdgcn_s_memrealtime();
#define NOW() ((unsigned)(__builtin_amdgcn_s_memrealtime() - tstart))
#define UNTIL(t_) while (NOW() < ((t_) >> 8)) __builtin_amdgcn_s_sleep(1)

    if (__builtin_expect(j < 16, 1)) {
        const int ks = ((b & 7) - c8) & 7, h = j >> 3;
        const bool split = ks == 5;
        if (wave > 8 || (wave > 4 && !split)) return;
        if (wave == 4) {
            f32x4 v[16];
            UNTIL(NS(W_DELAY));
            const float* src = p.w + (size_t)(32 * ks + 8 * (lane >> 4)) * NN + 64 * (c8 + 8 * h) + 4 * (lane & 15);
#pragma unroll
            for (int q = 0; q < 16; ++q) v[q] = *(const f32x4*)(src + (size_t)(q & 7) * NN + (q >> 3) * 1024);
            const unsigned call = (unsigned)__builtin_amdgcn_readfirstlane((int)__hip_atomic_load(cnt, __ATOMIC_RELAXED, __HIP_MEMORY_SCOPE_AGENT)) + 1u;
            const unsigned tag = ((((call - 1u) >> 1) + 1u) & 1u) ? TAG16 : 0u;
            const int cd = code_of<16>(v);
            const int sh = 126 - cd;
            char* dl = smem + L_RED + ((lane & 15) >> 2) * 1024 + ((lane >> 4) * 16 + 4 * (lane & 3)) * 16;
#pragma unroll
            for (int uh = 0; uh < 2; ++uh)
#pragma unroll
                for (int c = 0; c < 4; ++c) {
                    u32x4 fr;
#pragma unroll
                    for (int q = 0; q < 4; ++q)
                        fr[q] = (cd == 255 ? 0u : pack2(__builtin_ldexpf(v[uh * 8 + 2 * q][c], sh), __builtin_ldexpf(v[uh * 8 + 2 * q + 1][c], sh))) | tag |
                                ((((unsigned)cd >> (4 * (c & 1) + q)) & 1u) << 14);
                    *(u32x4*)(dl + uh * 4096 + c * 16) = fr;
                }
            asm volatile("s_waitcnt lgkmcnt(0)" ::: "memory");
            __builtin_amdgcn_s_barrier();
            if (lane == 0) __hip_atomic_store(cnt, call, __ATOMIC_RELAXED, __HIP_MEMORY_SCOPE_AGENT);
            return;
        }
        const bool own = ks < 5 || (split && wave < 4);
        const bool act = !split || (((lane >> 4) < SPL) == own);
        const int dgrp = own ? grp : grp ^ 1;
        const int dw = wave < 4 ? wave : wave - 5;
        const int dl = (gi + 1 + dw) & 3;
        const int tsl = ks < 5 ? 5 * gi + ks : (split ? 28 + gi : 20 + 2 * gi + (ks - 6));
        f32x4 v[4];
        {
            const int row0 = (c8 < 4 ? MS * ((dgrp << 2) | dl) + 64 * c8 : MS * (((dgrp ^ 1) << 2) | (c8 - 4)) + 64 * dl) + 32 * h;
            const float* xs = px + (size_t)(row0 + (lane & 15)) * KS + 32 * ks + 8 * (lane >> 4);
#pragma unroll
            for (int a = 0; a < 2; ++a) {
                v[2 * a] = *(const f32x4*)(xs + (size_t)a * 16 * KS);
                v[2 * a + 1] = *(const f32x4*)(xs + (size_t)a * 16 * KS + 4);
            }
        }
        unsigned call = __hip_atomic_load(cnt, __ATOMIC_RELAXED, __HIP_MEMORY_SCOPE_AGENT) + 1u;
        char* dd = p.rel[own ? 0 : 1][dw];
        call = (unsigned)__builtin_amdgcn_readfirstlane((int)call);
        const unsigned tag = ((((call - 1u) >> 1) + 1u) & 1u) ? TAG16 : 0u;
        dd += (size_t)(call & 1u) * BUF_BYTES + lane * 16;
        const int cd = code_of<4>(v);
        const int sh = 126 - cd;
        const unsigned nib = (unsigned)cd >> (4 * (lane & 1));
        const u32x4 xtag = {tag | ((nib & 1u) << 14), tag | ((nib & 2u) << 13), tag | ((nib & 4u) << 12), tag | ((nib & 8u) << 11)};
        u32x4 xf[2];
#pragma unroll
        for (int a = 0; a < 2; ++a) {
            xf[a][0] = pack2(__builtin_ldexpf(v[2 * a][0], sh), __builtin_ldexpf(v[2 * a][1], sh));
            xf[a][1] = pack2(__builtin_ldexpf(v[2 * a][2], sh), __builtin_ldexpf(v[2 * a][3], sh));
            xf[a][2] = pack2(__builtin_ldexpf(v[2 * a + 1][0], sh), __builtin_ldexpf(v[2 * a + 1][1], sh));
            xf[a][3] = pack2(__builtin_ldexpf(v[2 * a + 1][2], sh), __builtin_ldexpf(v[2 * a + 1][3], sh));
            if (cd == 255) xf[a] = u32x4{0u, 0u, 0u, 0u};
            xf[a] |= xtag;
        }
        const unsigned dq = own ? NS(P_MAJ) / 16 : NS(P_MIN) / 16;
        unsigned t00 = min(NOW(), 1u << 22) << 8;
        t00 = t00 > NS(T_OFS) ? t00 : NS(T_OFS);
        char* xd = dd + X_OFF + (size_t)(tsl * 32 + 4 * c8 + 2 * h) * 1024;
        const bool early = split && own && c8 < 4;
        if (early) { __builtin_amdgcn_s_barrier(); asm volatile("" ::: "memory"); }
        const unsigned tx = t00 + dq * (unsigned)(((c8 ^ 4) * 2 + h) * 2);
        UNTIL(tx);
        if (act) st16_sys(xd, xf[0]);
        UNTIL(tx + dq);
        if (act) st16_sys(xd + 1024, xf[1]);
        if (!early) { __builtin_amdgcn_s_barrier(); asm volatile("" ::: "memory"); }
        char* wd = dd + W_OFF + (size_t)(tsl * 32 + c8 + 8 * h) * 4096;
        const char* ls = smem + L_RED + lane * 16;
        unsigned due = t00 + dq * (unsigned)(32 + c8 * 2 + h);
#pragma unroll 1
        for (int s = 0; s < 8; ++s) {
            const u32x4 fr = *(const u32x4*)(ls + (s & 1) * 4096 + (s >> 1) * 1024);
            UNTIL(due);
            due += 16 * dq;
            if (act) st16_sys(wd + (s & 1) * 16 * 4096 + (s >> 1) * 1024, fr);
        }
        return;
    }

    const int jc = j - 16, u0 = jc & 1, kind = 1 - ((jc >> 1) & 1), rq = jc >> 2;
    unsigned call = __hip_atomic_load(cnt, __ATOMIC_RELAXED, __HIP_MEMORY_SCOPE_AGENT) + 1u;
    if (wave == 9 && lane < 17) s_misc[lane] = 0;
    call = (unsigned)__builtin_amdgcn_readfirstlane((int)call);
    __syncthreads();
    if (tid == 0) __hip_atomic_store(cnt, call, __ATOMIC_RELAXED, __HIP_MEMORY_SCOPE_AGENT);
    const bool ph = ((((call - 1u) >> 1) + 1u) & 1u) != 0u;
    const size_t boff = (size_t)(call & 1u) * BUF_BYTES;
    Spin sp; sp.t0 = tstart; sp.it = 0; sp.call = call; sp.tmo = (unsigned*)(me + T_OFF); sp.dead = s_misc + 16;
    bool dead = false;
#define LDS_WAIT(idx, n) while (!dead && __builtin_amdgcn_readfirstlane(s_misc[idx]) < (n)) { if (sp.fail_lds()) dead = true; }
#define LDS_SIGNAL(idx) do { if (lane == 0) __hip_atomic_fetch_add((lds_int*)(s_misc + (idx)), 1, __ATOMIC_RELAXED, __HIP_MEMORY_SCOPE_WORKGROUP); } while (0)

    if (wave >= 8) {
        clean_vm();
        const int a = wave & 3;
        const unsigned tag = ph ? TAG32 : 0u;
        char* pp = (kind ? p.rel[1][(rq - gi - 1) & 3] : me) + boff + P_OFF + (size_t)((c8 * 2 + u0) * 4 + (kind ? gi : rq)) * 32768 + a * 1024 + lane * 16;
        float* op = p.out + (size_t)(64 * rq + 16 * a + (lane & 15)) * NN + 64 * (c8 + 8 * u0) + 4 * (lane >> 4);
#pragma unroll 1
        for (int s = (wave >> 2) & 1; s < 8; s += 2) {
            LDS_WAIT(s, 8);
            asm volatile("" ::: "memory");
            const char* rp = smem + L_RED + (s & 3) * 32768 + a * 1024 + lane * 16;
            f32x4 sum = *(const f32x4*)rp;
#pragma unroll
            for (int w = 1; w < 8; ++w) sum += *(const f32x4*)(rp + w * 4096);
            asm volatile("" : "+v"(sum) :: "memory");
            LDS_SIGNAL(8 + s);
            const unsigned tp = NS(T_OFS + 10 * P_MIN + T_PART) + NS(P_PART) / 16 * (unsigned)(s * 16 + c8 * 2 + u0);
            if (kind) {
                UNTIL(tp);
                if (!dead) st16_sys(pp + s * 4096, (__builtin_bit_cast(u32x4, sum) & ~TAG32) | tag);
            } else {
                UNTIL(tp + NS(T_PL));
                u32x4 pv = {0u, 0u, 0u, 0u};
                while (!dead) {
                    asm volatile("global_load_dwordx4 %0, %1, off sc0 sc1\n\ts_waitcnt vmcnt(0)" : "=&v"(pv) : "v"(pp + s * 4096) : "memory");
                    pv ^= tag;
                    if (__ballot((or4(pv) & TAG32) != 0u) == 0ull) break;
                    if (sp.fail()) dead = true;
                }
                const f32x4 y = sum + __builtin_bit_cast(f32x4, pv);
                f32x4 o;
#pragma unroll
                for (int r = 0; r < 4; ++r) {
                    const float c = fminf(fmaxf(y[r], -60.f), 60.f);
                    o[r] = y[r] / (1.f + __expf(-c));
                }
                if (dead) o = f32x4{__builtin_nanf(""), __builtin_nanf(""), __builtin_nanf(""), __builtin_nanf("")};
                st16_sys((char*)(op + (s & 1) * 1024 + (s >> 1) * 16), __builtin_bit_cast(u32x4, o));
            }
        }
        return;
    }

    const int t0 = wave * 4;
    const unsigned tag = ph ? TAG16 : 0u;
    const bool fast = wave == 5 || wave == 6;
    const unsigned dq = fast ? NS(P_MIN) / 16 : NS(P_MAJ) / 16;
    __amdgpu_buffer_rsrc_t xr = __builtin_amdgcn_make_buffer_rsrc((void*)(me + boff + X_OFF + (size_t)(t0 * 32 + 16 * kind + 4 * rq) * 1024), 0, 4 * 32 * 1024, 0x00020000);
    UNTIL(NS(T_OFS + T_LANDX) + dq * (unsigned)((kind ? 0 : 16) + 4 * rq + 3));
    while (!dead) {
        const unsigned v = __builtin_amdgcn_raw_buffer_load_b32(xr, (lane & 3) * 32768 + 3 * 1024 + ((wave == 7 && (lane & 4)) ? SPL * 256 - 4 : 1020), 0, 17) ^ tag;
        if (__ballot((v & TAG16) != 0u) == 0ull) break;
        __builtin_amdgcn_s_sleep(12);
        if (sp.fail()) dead = true;
    }
    u32x4 af[4][4];
    while (!dead) {
        unsigned m = 0u;
#pragma unroll
        for (int tt = 0; tt < 4; ++tt)
#pragma unroll
            for (int a = 0; a < 4; ++a)
                af[tt][a] = __builtin_amdgcn_raw_buffer_load_b128(xr, lane * 16, tt * 32768 + a * 1024, 17);
#pragma unroll
        for (int tt = 0; tt < 4; ++tt)
#pragma unroll
            for (int a = 0; a < 4; ++a) { af[tt][a] ^= tag; m |= or4(af[tt][a]); }
        if (__ballot((m & TAG16) != 0u) == 0ull) break;
        __builtin_amdgcn_s_sleep(12);
        if (sp.fail()) dead = true;
    }
    clean_vm();
    int cx[4][2], ex[2] = {0, 0};
#pragma unroll
    for (int tt = 0; tt < 4; ++tt) {
        cx[tt][0] = code_from(af[tt][0], lane); cx[tt][1] = code_from(af[tt][2], lane);
#pragma unroll
        for (int a = 0; a < 4; ++a) af[tt][a] &= PAYLOAD;
    }
    const char* wq = me + boff + W_OFF + (size_t)(t0 * 32 + c8 + 8 * u0) * 4096 + lane * 16;
    unsigned due = NS(T_OFS + T_LAND) + (fast ? NS(300) : (kind ? 0u : NS(T_OWN))) + dq * (unsigned)(32 + c8 * 2 + u0);
    unsigned lag = 0;
#pragma unroll 1
    for (int s = 0; s < 8; ++s) {
        const unsigned g = due >> 8;
        due += 16 * dq;
        while (NOW() < g + lag) __builtin_amdgcn_s_sleep(1);
        lag = lag > LAG_DEC / 10 ? lag - LAG_DEC / 10 : 0;
        const char* q = wq + (s & 1) * (16 * 4096) + (s >> 1) * 1024;
        u32x4 cur[4];
        for (;;) {
            const unsigned t_try = NOW();
            asm volatile("global_load_dwordx4 %0, %4, off sc0 sc1\n\tglobal_load_dwordx4 %1, %5, off sc0 sc1\n\t"
                         "global_load_dwordx4 %2, %6, off sc0 sc1\n\tglobal_load_dwordx4 %3, %7, off sc0 sc1\n\t"
                         "s_waitcnt vmcnt(0)"
                         : "=&v"(cur[0]), "=&v"(cur[1]), "=&v"(cur[2]), "=&v"(cur[3])
                         : "v"(q), "v"(q + 32 * 4096), "v"(q + 2 * 32 * 4096), "v"(q + 3 * 32 * 4096) : "memory");
#pragma unroll
            for (int tt = 0; tt < 4; ++tt) cur[tt] ^= tag;
            if (dead || __ballot(((or4(cur[0]) | or4(cur[1]) | or4(cur[2]) | or4(cur[3])) & TAG16) != 0u) == 0ull) break;
            lag = t_try + LAG_INC / 10 - g;
            __builtin_amdgcn_s_sleep(6);
            if (sp.fail()) { dead = true; break; }
        }
        if (s == 0) {
            bool bad = false, same = true;
#pragma unroll
            for (int tt = 0; tt < 4; ++tt) {
                const int cw = code_from(cur[tt], lane);
                bad |= cw == 255 || cx[tt][0] == 255 || cx[tt][1] == 255;
                cx[tt][0] += cw; cx[tt][1] += cw;
            }
            ex[0] = __builtin_amdgcn_readfirstlane(cx[0][0]); ex[1] = __builtin_amdgcn_readfirstlane(cx[0][1]);
#pragma unroll
            for (int tt = 0; tt < 4; ++tt) same &= cx[tt][0] == ex[0] && cx[tt][1] == ex[1];
            if (__ballot(bad || !same) != 0ull) {
                const bool poison = __ballot(bad) != 0ull;
#pragma unroll
                for (int hh = 0; hh < 2; ++hh) {
                    ex[hh] = (int)wave_max_u((unsigned)max(max(cx[0][hh], cx[1][hh]), max(cx[2][hh], cx[3][hh])));
#pragma unroll
                    for (int tt = 0; tt < 4; ++tt) {
                        const _Float16 m = poison ? (_Float16)__builtin_nanf("") : (_Float16)__builtin_ldexpf(1.f, cx[tt][hh] - ex[hh]);
#pragma unroll
                        for (int a = 0; a < 2; ++a) {
                            f16x8 t = __builtin_bit_cast(f16x8, af[tt][2 * hh + a]);
                            t *= m;
                            af[tt][2 * hh + a] = __builtin_bit_cast(u32x4, t);
                        }
                    }
                }
            }
            ex[0] -= 252; ex[1] -= 252;
        }
#pragma unroll
        for (int tt = 0; tt < 4; ++tt) cur[tt] &= PAYLOAD;
        f32x4 acc[4];
#pragma unroll
        for (int a = 0; a < 4; ++a) acc[a] = f32x4{0.f, 0.f, 0.f, 0.f};
#pragma unroll
        for (int tt = 0; tt < 4; ++tt)
#pragma unroll
            for (int a = 0; a < 4; ++a)
                acc[a] = __builtin_amdgcn_mfma_f32_16x16x32_f16(__builtin_bit_cast(f16x8, cur[tt]), __builtin_bit_cast(f16x8, af[tt][a]), acc[a], 0, 0, 0);
        if (s >= 4) LDS_WAIT(8 + s - 4, 4);
        asm volatile("" ::: "memory");
        char* rp = smem + L_RED + (s & 3) * 32768 + wave * 4096 + lane * 16;
#pragma unroll
        for (int a = 0; a < 4; ++a) {
            const int e = ex[a >> 1];
            *(f32x4*)(rp + a * 1024) = f32x4{__builtin_ldexpf(acc[a][0], e), __builtin_ldexpf(acc[a][1], e),
                                             __builtin_ldexpf(acc[a][2], e), __builtin_ldexpf(acc[a][3], e)};
        }
        asm volatile("" ::: "memory");
        LDS_SIGNAL(s);
    }
}

}

extern "C" void kernel_launch(int rank, int nranks, ncclComm_t comm,
                              hipStream_t stream,
                              void* const* d_in, const int* in_sizes, int n_in,
                              void* d_out, int out_size,
                              const int* in_dtypes, int out_dtype,
                              void* const* d_out_all,
                              void* const* d_ws_all, size_t ws_bytes) {
    if (nranks != NR || (unsigned)rank >= (unsigned)NR || ws_bytes < WS_NEED || n_in < 2) {
        fprintf(stderr, "unexpected configuration\n");
        return;
    }
    Params p;
    p.x = (const float*)d_in[0];
    p.w = (const float*)d_in[1];
    p.out = (float*)d_out;
    p.me = (char*)d_ws_all[rank];
    for (int g = 0; g < 2; ++g)
        for (int d = 0; d < 4; ++d) p.rel[g][d] = (char*)d_ws_all[(((rank >> 2) ^ g) << 2) | ((rank + 1 + d) & 3)];
    p.rank = rank;
    hipLaunchKernelGGL(fused_kernel, dim3(NWG), dim3(NTHREADS), 0, stream, p);
    HIP_CHECK(hipGetLastError());
}
```

```cpp
#include <hip/hip_runtime.h>
#include <rccl/rccl.h>
#include <cstdio>

#define HIP_CHECK(expr) do { hipError_t _e = (expr); if (_e != hipSuccess) { \
    fprintf(stderr, "HIP error %s at %s:%d\n", hipGetErrorString(_e), __FILE__, __LINE__); \
    return; } } while (0)

#define T_OFS 1000
#define P_MAJ 1465
#define P_MIN 768
#define P_PART 1125
#define T_PART 300
#define T_LAND 600
#define T_LANDX 900
#define T_OWN 200
#define T_PL 400
#define W_DELAY 500
#define LAG_INC 400
#define LAG_DEC 100

namespace {

constexpr int NR = 8;
constexpr int NN = 2048;
constexpr int KS = 256;
constexpr int MS = 256;
constexpr int SPL = 1;
constexpr int NWG = 256;
constexpr int NTHREADS = 1024;

constexpr unsigned E_MAJ = 5 * 1024 + SPL * 256;
constexpr unsigned E_MIN = 2 * 1024 + (4 - SPL) * 256;
constexpr unsigned R_MAJ = 160 * E_MAJ, R_MIN = 160 * E_MIN;
constexpr size_t P_OFF = 5ull << 20;
constexpr size_t BUF_BYTES = 7ull << 20;
static_assert(4 * (size_t)(R_MAJ + R_MIN) <= P_OFF, "layout");
constexpr size_t C_OFF = 2 * BUF_BYTES;
constexpr size_t T_OFF = C_OFF + NWG * 4;
constexpr size_t WS_NEED = T_OFF + 128;
constexpr unsigned long long DEADLINE = 150000000ull;
constexpr unsigned TAG16 = 0x40000000u;
constexpr unsigned PAYLOAD = 0xBFFFBFFFu;
constexpr unsigned TAG32 = 1u;
constexpr unsigned NS(int ns) { return (unsigned)ns * 256u / 10u; }

typedef __attribute__((address_space(3))) volatile int lds_vint;
typedef __attribute__((address_space(3))) int lds_int;
typedef float f32x4 __attribute__((ext_vector_type(4)));
typedef unsigned u32x4 __attribute__((ext_vector_type(4)));
typedef _Float16 f16x8 __attribute__((ext_vector_type(8)));
typedef _Float16 f16x2 __attribute__((ext_vector_type(2)));

struct Params {
    const float* x;
    const float* w;
    float* out;
    char* me;
    char* rel[2][4];
    int rank;
};

__device__ __forceinline__ void st16_sys(char* p, u32x4 v) {
    asm volatile("global_store_dwordx4 %0, %1, off sc0 sc1\n\ts_nop 1" :: "v"(p), "v"(v) : "memory");
}
__device__ __forceinline__ void clean_vm() { __builtin_amdgcn_s_waitcnt(0x0F70); }

__device__ __forceinline__ unsigned pack2(float a, float b) {
    f16x2 h = {(_Float16)a, (_Float16)b};
    return __builtin_bit_cast(unsigned, h);
}
__device__ __forceinline__ unsigned or4(u32x4 v) { return (v[0] | v[1]) | (v[2] | v[3]); }

__device__ __forceinline__ unsigned wave_max_u(unsigned v) {
#define DPP_MAX(ctrl, rmask) do { const unsigned t_ = (unsigned)__builtin_amdgcn_update_dpp(0, (int)v, ctrl, rmask, 0xf, false); v = v > t_ ? v : t_; } while (0)
    DPP_MAX(0x111, 0xf); DPP_MAX(0x112, 0xf); DPP_MAX(0x114, 0xf); DPP_MAX(0x118, 0xf);
    DPP_MAX(0x142, 0xa); DPP_MAX(0x143, 0xc);
    return (unsigned)__builtin_amdgcn_readlane((int)v, 63);
}
template <int N> __device__ __forceinline__ int code_of(const f32x4* v) {
    unsigned am = 0;
#pragma unroll
    for (int j = 0; j < N; ++j) {
        const u32x4 u = __builtin_bit_cast(u32x4, v[j]) & 0x7fffffffu;
#pragma unroll
        for (int f = 0; f < 4; ++f) am = u[f] > am ? u[f] : am;
    }
    const int e = (int)(wave_max_u(am) >> 23);
    return e == 255 ? 255 : ((e + 1) | 3) - 1;
}
__device__ __forceinline__ int code_from(u32x4 v, int lane) {
    const int nib = (int)(((v[0] >> 14) & 1u) | ((v[1] >> 13) & 2u) | ((v[2] >> 12) & 4u) | ((v[3] >> 11) & 8u));
    const int oth = __shfl_xor(nib, 1);
    return (lane & 1) ? (nib << 4) | oth : (oth << 4) | nib;
}

struct Spin {
    unsigned long long t0; unsigned it; unsigned call; unsigned* tmo; lds_vint* dead;
    __device__ __forceinline__ bool give_up() {
        __hip_atomic_store(tmo, call, __ATOMIC_RELAXED, __HIP_MEMORY_SCOPE_AGENT);
        *dead = 1;
        return true;
    }
    __device__ __forceinline__ bool fail_lds() {
        __builtin_amdgcn_s_sleep(1);
        asm volatile("" ::: "memory");
        if (__builtin_amdgcn_readfirstlane(*dead) != 0) return true;
        if ((++it & 1023u) == 0u && __builtin_amdgcn_s_memrealtime() - t0 > DEADLINE) return give_up();
        return false;
    }
    __device__ __forceinline__ bool fail() {
        __builtin_amdgcn_s_sleep(2);
        asm volatile("" ::: "memory");
        if ((++it & 63u) != 0u) return false;
        if (__builtin_amdgcn_readfirstlane(*dead)) return true;
        if ((unsigned)__builtin_amdgcn_readfirstlane((int)__hip_atomic_load(tmo, __ATOMIC_RELAXED, __HIP_MEMORY_SCOPE_AGENT)) == call ||
            __builtin_amdgcn_s_memrealtime() - t0 > DEADLINE) return give_up();
        return false;
    }
};

constexpr int L_RED = 0;
constexpr int L_MISC = 131072;
constexpr int L_BYTES = L_MISC + 128;

__global__ void __launch_bounds__(NTHREADS, 1) fused_kernel(Params p) {
    __shared__ __attribute__((aligned(16))) char smem[L_BYTES];
    lds_vint* s_misc = (lds_vint*)(smem + L_MISC);

    const int tid = threadIdx.x, lane = tid & 63;
    const int wave = __builtin_amdgcn_readfirstlane(tid >> 6);
    const int b = blockIdx.x;
    const int rank = p.rank, grp = rank >> 2, gi = rank & 3;
    char* me = p.me;
    const float* px = p.x;
    asm volatile("" :: "s"(px));
    const int j = b >> 3;
    const int c8 = j < 16 ? (j & 7) : (b & 7);
    unsigned* cnt = (unsigned*)(me + C_OFF) + b;
    const unsigned long long tstart = __builtin_amdgcn_s_memrealtime();
#define NOW() ((unsigned)(__builtin_amdgcn_s_memrealtime() - tstart))
#define UNTIL(t_) while (NOW() < ((t_) >> 8)) __builtin_amdgcn_s_sleep(1)

    if (__builtin_expect(j < 16, 1)) {
        const int ks = ((b & 7) - c8) & 7, h = j >> 3;
        const bool split = ks == 5;
        if (wave > 8 || (wave > 4 && !split)) return;
        if (wave == 4) {
            f32x4 v[16];
            UNTIL(NS(W_DELAY));
            const float* src = p.w + (size_t)(32 * ks + 8 * (lane >> 4)) * NN + 64 * (c8 + 8 * h) + 4 * (lane & 15);
#pragma unroll
            for (int q = 0; q < 16; ++q) v[q] = *(const f32x4*)(src + (size_t)(q & 7) * NN + (q >> 3) * 1024);
            const unsigned call = (unsigned)__builtin_amdgcn_readfirstlane((int)__hip_atomic_load(cnt, __ATOMIC_RELAXED, __HIP_MEMORY_SCOPE_AGENT)) + 1u;
            const unsigned tag = ((((call - 1u) >> 1) + 1u) & 1u) ? TAG16 : 0u;
            const int cd = code_of<16>(v);
            const int sh = 126 - cd;
            char* dl = smem + L_RED + ((lane & 15) >> 2) * 1024 + ((lane >> 4) * 16 + 4 * (lane & 3)) * 16;
#pragma unroll
            for (int uh = 0; uh < 2; ++uh)
#pragma unroll
                for (int c = 0; c < 4; ++c) {
                    u32x4 fr;
#pragma unroll
                    for (int q = 0; q < 4; ++q)
                        fr[q] = (cd == 255 ? 0u : pack2(__builtin_ldexpf(v[uh * 8 + 2 * q][c], sh), __builtin_ldexpf(v[uh * 8 + 2 * q + 1][c], sh))) | tag |
                                ((((unsigned)cd >> (4 * (c & 1) + q)) & 1u) << 14);
                    *(u32x4*)(dl + uh * 4096 + c * 16) = fr;
                }
            asm volatile("s_waitcnt lgkmcnt(0)" ::: "memory");
            __builtin_amdgcn_s_barrier();
            if (lane == 0) __hip_atomic_store(cnt, call, __ATOMIC_RELAXED, __HIP_MEMORY_SCOPE_AGENT);
            return;
        }
        const bool own = ks < 5 || (split && wave < 4);
        const bool act = !split || (((lane >> 4) < SPL) == own);
        const int dgrp = own ? grp : grp ^ 1;
        const int dw = wave < 4 ? wave : wave - 5;
        const int dl = (gi + 1 + dw) & 3;
        f32x4 v[4];
        {
            const int row0 = (c8 < 4 ? MS * ((dgrp << 2) | dl) + 64 * c8 : MS * (((dgrp ^ 1) << 2) | (c8 - 4)) + 64 * dl) + 32 * h;
            const float* xs = px + (size_t)(row0 + (lane & 15)) * KS + 32 * ks + 8 * (lane >> 4);
#pragma unroll
            for (int a = 0; a < 2; ++a) {
                v[2 * a] = *(const f32x4*)(xs + (size_t)a * 16 * KS);
                v[2 * a + 1] = *(const f32x4*)(xs + (size_t)a * 16 * KS + 4);
            }
        }
        unsigned call = __hip_atomic_load(cnt, __ATOMIC_RELAXED, __HIP_MEMORY_SCOPE_AGENT) + 1u;
        char* dd = p.rel[own ? 0 : 1][dw];
        call = (unsigned)__builtin_amdgcn_readfirstlane((int)call);
        const unsigned tag = ((((call - 1u) >> 1) + 1u) & 1u) ? TAG16 : 0u;
        const unsigned esz = own ? E_MAJ : E_MIN;
        dd += (size_t)(call & 1u) * BUF_BYTES + lane * 16 +
              (own ? gi * R_MAJ + ks * 1024 : 4 * R_MAJ + gi * R_MIN + (split ? 2048 - SPL * 256 : (ks - 6) * 1024));
        const int cd = code_of<4>(v);
        const int sh = 126 - cd;
        const unsigned nib = (unsigned)cd >> (4 * (lane & 1));
        const u32x4 xtag = {tag | ((nib & 1u) << 14), tag | ((nib & 2u) << 13), tag | ((nib & 4u) << 12), tag | ((nib & 8u) << 11)};
        u32x4 xf[2];
#pragma unroll
        for (int a = 0; a < 2; ++a) {
            xf[a][0] = pack2(__builtin_ldexpf(v[2 * a][0], sh), __builtin_ldexpf(v[2 * a][1], sh));
            xf[a][1] = pack2(__builtin_ldexpf(v[2 * a][2], sh), __builtin_ldexpf(v[2 * a][3], sh));
            xf[a][2] = pack2(__builtin_ldexpf(v[2 * a + 1][0], sh), __builtin_ldexpf(v[2 * a + 1][1], sh));
            xf[a][3] = pack2(__builtin_ldexpf(v[2 * a + 1][2], sh), __builtin_ldexpf(v[2 * a + 1][3], sh));
            if (cd == 255) xf[a] = u32x4{0u, 0u, 0u, 0u};
            xf[a] |= xtag;
        }
        const unsigned dq = own ? NS(P_MAJ) / 16 : NS(P_MIN) / 16;
        unsigned t00 = min(NOW(), 1u << 22) << 8;
        t00 = t00 > NS(T_OFS) ? t00 : NS(T_OFS);
        const unsigned xs0 = (unsigned)(((c8 ^ 4) * 2 + h) * 2);
        const bool early = split && own && c8 < 4;
        if (early) { __builtin_amdgcn_s_barrier(); asm volatile("" ::: "memory"); }
        const unsigned tx = t00 + dq * xs0;
        UNTIL(tx);
        if (act) st16_sys(dd + xs0 * esz, xf[0]);
        UNTIL(tx + dq);
        if (act) st16_sys(dd + (xs0 + 1) * esz, xf[1]);
        if (!early) { __builtin_amdgcn_s_barrier(); asm volatile("" ::: "memory"); }
        char* wd = dd + (unsigned)(32 + c8 * 2 + h) * esz;
        const char* ls = smem + L_RED + lane * 16;
        unsigned due = t00 + dq * (unsigned)(32 + c8 * 2 + h);
#pragma unroll 1
        for (int s = 0; s < 8; ++s) {
            const u32x4 fr = *(const u32x4*)(ls + (s & 1) * 4096 + (s >> 1) * 1024);
            UNTIL(due);
            due += 16 * dq;
            if (act) st16_sys(wd + (unsigned)s * 16u * esz, fr);
        }
        return;
    }

    const int jc = j - 16, u0 = jc & 1, kind = 1 - ((jc >> 1) & 1), rq = jc >> 2;
    unsigned call = __hip_atomic_load(cnt, __ATOMIC_RELAXED, __HIP_MEMORY_SCOPE_AGENT) + 1u;
    if (wave == 9 && lane < 17) s_misc[lane] = 0;
    call = (unsigned)__builtin_amdgcn_readfirstlane((int)call);
    __syncthreads();
    if (tid == 0) __hip_atomic_store(cnt, call, __ATOMIC_RELAXED, __HIP_MEMORY_SCOPE_AGENT);
    const bool ph = ((((call - 1u) >> 1) + 1u) & 1u) != 0u;
    const size_t boff = (size_t)(call & 1u) * BUF_BYTES;
    Spin sp; sp.t0 = tstart; sp.it = 0; sp.call = call; sp.tmo = (unsigned*)(me + T_OFF); sp.dead = s_misc + 16;
    bool dead = false;
#define LDS_WAIT(idx, n) while (!dead && __builtin_amdgcn_readfirstlane(s_misc[idx]) < (n)) { if (sp.fail_lds()) dead = true; }
#define LDS_SIGNAL(idx) do { if (lane == 0) __hip_atomic_fetch_add((lds_int*)(s_misc + (idx)), 1, __ATOMIC_RELAXED, __HIP_MEMORY_SCOPE_WORKGROUP); } while (0)

    if (wave >= 8) {
        clean_vm();
        const int a = wave & 3;
        const unsigned tag = ph ? TAG32 : 0u;
        char* pp = (kind ? p.rel[1][(rq - gi - 1) & 3] : me) + boff + P_OFF + (size_t)(kind ? gi : rq) * 524288 + (c8 * 2 + u0) * 4096 + a * 1024 + lane * 16;
        float* op = p.out + (size_t)(64 * rq + 16 * a + (lane & 15)) * NN + 64 * (c8 + 8 * u0) + 4 * (lane >> 4);
#pragma unroll 1
        for (int s = (wave >> 2) & 1; s < 8; s += 2) {
            LDS_WAIT(s, 8);
            asm volatile("" ::: "memory");
            const char* rp = smem + L_RED + (s & 3) * 32768 + a * 1024 + lane * 16;
            f32x4 sum = *(const f32x4*)rp;
#pragma unroll
            for (int w = 1; w < 8; ++w) sum += *(const f32x4*)(rp + w * 4096);
            asm volatile("" : "+v"(sum) :: "memory");
            LDS_SIGNAL(8 + s);
            const unsigned tp = NS(T_OFS + 10 * P_MIN + T_PART) + NS(P_PART) / 16 * (unsigned)(s * 16 + c8 * 2 + u0);
            if (kind) {
                UNTIL(tp);
                if (!dead) st16_sys(pp + s * 65536, (__builtin_bit_cast(u32x4, sum) & ~TAG32) | tag);
            } else {
                UNTIL(tp + NS(T_PL));
                u32x4 pv = {0u, 0u, 0u, 0u};
                while (!dead) {
                    asm volatile("global_load_dwordx4 %0, %1, off sc0 sc1\n\ts_waitcnt vmcnt(0)" : "=&v"(pv) : "v"(pp + s * 65536) : "memory");
                    pv ^= tag;
                    if (__ballot((or4(pv) & TAG32) != 0u) == 0ull) break;
                    if (sp.fail()) dead = true;
                }
                const f32x4 y = sum + __builtin_bit_cast(f32x4, pv);
                f32x4 o;
#pragma unroll
                for (int r = 0; r < 4; ++r) {
                    const float c = fminf(fmaxf(y[r], -60.f), 60.f);
                    o[r] = y[r] / (1.f + __expf(-c));
                }
                if (dead) o = f32x4{__builtin_nanf(""), __builtin_nanf(""), __builtin_nanf(""), __builtin_nanf("")};
                st16_sys((char*)(op + (s & 1) * 1024 + (s >> 1) * 16), __builtin_bit_cast(u32x4, o));
            }
        }
        return;
    }

    const int t0 = wave * 4;
    const unsigned tag = ph ? TAG16 : 0u;
    const bool fast = wave == 5 || wave == 6;
    const unsigned dq = fast ? NS(P_MIN) / 16 : NS(P_MAJ) / 16;
    const bool lmaj = wave < 5 || (wave == 7 && (lane >> 4) < SPL);
    const unsigned esz = lmaj ? E_MAJ : E_MIN;
    unsigned fo[4];
#pragma unroll
    for (int tt = 0; tt < 4; ++tt) {
        const int t = t0 + tt;
        fo[tt] = lane * 16 + (wave < 5 ? (t / 5) * R_MAJ + (t % 5) * 1024
                              : wave < 7 ? 4 * R_MAJ + ((t - 20) >> 1) * R_MIN + ((t - 20) & 1) * 1024
                              : lmaj ? tt * R_MAJ + 5120 : 4 * R_MAJ + tt * R_MIN + 2048 - SPL * 256);
    }
    __amdgpu_buffer_rsrc_t xr = __builtin_amdgcn_make_buffer_rsrc((void*)(me + boff), 0, 5 << 20, 0x00020000);
    const unsigned xs0 = (unsigned)((kind ? 0 : 16) + 4 * rq) * esz;
    UNTIL(NS(T_OFS + T_LANDX) + dq * (unsigned)((kind ? 0 : 16) + 4 * rq + 3));
    while (!dead) {
        unsigned v = tag;
        if ((lane & 12) == 12 && ((lane >> 4) == 3 || (lane >> 4) == SPL - 1)) {
            const int tt = lane & 3;
            v = __builtin_amdgcn_raw_buffer_load_b32(xr, (tt == 0 ? fo[0] : tt == 1 ? fo[1] : tt == 2 ? fo[2] : fo[3]) + xs0 + 3 * esz, 0, 17);
        }
        if (__ballot(((v ^ tag) & TAG16) != 0u) == 0ull) break;
        __builtin_amdgcn_s_sleep(12);
        if (sp.fail()) dead = true;
    }
    u32x4 af[4][4];
    while (!dead) {
        unsigned m = 0u;
#pragma unroll
        for (int tt = 0; tt < 4; ++tt)
#pragma unroll
            for (int a = 0; a < 4; ++a)
                af[tt][a] = __builtin_amdgcn_raw_buffer_load_b128(xr, fo[tt] + xs0 + a * esz, 0, 17);
#pragma unroll
        for (int tt = 0; tt < 4; ++tt)
#pragma unroll
            for (int a = 0; a < 4; ++a) { af[tt][a] ^= tag; m |= or4(af[tt][a]); }
        if (__ballot((m & TAG16) != 0u) == 0ull) break;
        __builtin_amdgcn_s_sleep(12);
        if (sp.fail()) dead = true;
    }
    clean_vm();
    int cx[4][2], ex[2] = {0, 0};
#pragma unroll
    for (int tt = 0; tt < 4; ++tt) {
        cx[tt][0] = code_from(af[tt][0], lane); cx[tt][1] = code_from(af[tt][2], lane);
#pragma unroll
        for (int a = 0; a < 4; ++a) af[tt][a] &= PAYLOAD;
    }
    const char* wq = me + boff;
#pragma unroll
    for (int tt = 0; tt < 4; ++tt) fo[tt] += (unsigned)(32 + c8 * 2 + u0) * esz;
    unsigned due = NS(T_OFS + T_LAND) + (fast ? NS(300) : (kind ? 0u : NS(T_OWN))) + dq * (unsigned)(32 + c8 * 2 + u0);
    unsigned lag = 0;
#pragma unroll 1
    for (int s = 0; s < 8; ++s) {
        const unsigned g = due >> 8;
        due += 16 * dq;
        while (NOW() < g + lag) __builtin_amdgcn_s_sleep(1);
        lag = lag > LAG_DEC / 10 ? lag - LAG_DEC / 10 : 0;
        u32x4 cur[4];
        for (;;) {
            const unsigned t_try = NOW();
            asm volatile("s_nop 4\n\tglobal_load_dwordx4 %0, %4, %8 sc0 sc1\n\tglobal_load_dwordx4 %1, %5, %8 sc0 sc1\n\t"
                         "global_load_dwordx4 %2, %6, %8 sc0 sc1\n\tglobal_load_dwordx4 %3, %7, %8 sc0 sc1\n\t"
                         "s_waitcnt vmcnt(0)"
                         : "=&v"(cur[0]), "=&v"(cur[1]), "=&v"(cur[2]), "=&v"(cur[3])
                         : "v"(fo[0]), "v"(fo[1]), "v"(fo[2]), "v"(fo[3]), "s"(wq) : "memory");
#pragma unroll
            for (int tt = 0; tt < 4; ++tt) cur[tt] ^= tag;
            if (dead || __ballot(((or4(cur[0]) | or4(cur[1]) | or4(cur[2]) | or4(cur[3])) & TAG16) != 0u) == 0ull) break;
            lag = t_try + LAG_INC / 10 - g;
            __builtin_amdgcn_s_sleep(6);
            if (sp.fail()) { dead = true; break; }
        }
#pragma unroll
        for (int tt = 0; tt < 4; ++tt) fo[tt] += 16u * esz;
        if (s == 0) {
            bool bad = false, same = true;
#pragma unroll
            for (int tt = 0; tt < 4; ++tt) {
                const int cw = code_from(cur[tt], lane);
                bad |= cw == 255 || cx[tt][0] == 255 || cx[tt][1] == 255;
                cx[tt][0] += cw; cx[tt][1] += cw;
            }
            ex[0] = __builtin_amdgcn_readfirstlane(cx[0][0]); ex[1] = __builtin_amdgcn_readfirstlane(cx[0][1]);
#pragma unroll
            for (int tt = 0; tt < 4; ++tt) same &= cx[tt][0] == ex[0] && cx[tt][1] == ex[1];
            if (__ballot(bad || !same) != 0ull) {
                const bool poison = __ballot(bad) != 0ull;
#pragma unroll
                for (int hh = 0; hh < 2; ++hh) {
                    ex[hh] = (int)wave_max_u((unsigned)max(max(cx[0][hh], cx[1][hh]), max(cx[2][hh], cx[3][hh])));
#pragma unroll
                    for (int tt = 0; tt < 4; ++tt) {
                        const _Float16 m = poison ? (_Float16)__builtin_nanf("") : (_Float16)__builtin_ldexpf(1.f, cx[tt][hh] - ex[hh]);
#pragma unroll
                        for (int a = 0; a < 2; ++a) {
                            f16x8 t = __builtin_bit_cast(f16x8, af[tt][2 * hh + a]);
                            t *= m;
                            af[tt][2 * hh + a] = __builtin_bit_cast(u32x4, t);
                        }
                    }
                }
            }
            ex[0] -= 252; ex[1] -= 252;
        }
#pragma unroll
        for (int tt = 0; tt < 4; ++tt) cur[tt] &= PAYLOAD;
        f32x4 acc[4];
#pragma unroll
        for (int a = 0; a < 4; ++a) acc[a] = f32x4{0.f, 0.f, 0.f, 0.f};
#pragma unroll
        for (int tt = 0; tt < 4; ++tt)
#pragma unroll
            for (int a = 0; a < 4; ++a)
                acc[a] = __builtin_amdgcn_mfma_f32_16x16x32_f16(__builtin_bit_cast(f16x8, cur[tt]), __builtin_bit_cast(f16x8, af[tt][a]), acc[a], 0, 0, 0);
        if (s >= 4) LDS_WAIT(8 + s - 4, 4);
        asm volatile("" ::: "memory");
        char* rp = smem + L_RED + (s & 3) * 32768 + wave * 4096 + lane * 16;
#pragma unroll
        for (int a = 0; a < 4; ++a) {
            const int e = ex[a >> 1];
            *(f32x4*)(rp + a * 1024) = f32x4{__builtin_ldexpf(acc[a][0], e), __builtin_ldexpf(acc[a][1], e),
                                             __builtin_ldexpf(acc[a][2], e), __builtin_ldexpf(acc[a][3], e)};
        }
        asm volatile("" ::: "memory");
        LDS_SIGNAL(s);
    }
}

}

extern "C" void kernel_launch(int rank, int nranks, ncclComm_t comm,
                              hipStream_t stream,
                              void* const* d_in, const int* in_sizes, int n_in,
                              void* d_out, int out_size,
                              const int* in_dtypes, int out_dtype,
                              void* const* d_out_all,
                              void* const* d_ws_all, size_t ws_bytes) {
    if (nranks != NR || (unsigned)rank >= (unsigned)NR || ws_bytes < WS_NEED || n_in < 2) {
        fprintf(stderr, "unexpected configuration\n");
        return;
    }
    Params p;
    p.x = (const float*)d_in[0];
    p.w = (const float*)d_in[1];
    p.out = (float*)d_out;
    p.me = (char*)d_ws_all[rank];
    for (int g = 0; g < 2; ++g)
        for (int d = 0; d < 4; ++d) p.rel[g][d] = (char*)d_ws_all[(((rank >> 2) ^ g) << 2) | ((rank + 1 + d) & 3)];
    p.rank = rank;
    hipLaunchKernelGGL(fused_kernel, dim3(NWG), dim3(NTHREADS), 0, stream, p);
    HIP_CHECK(hipGetLastError());
}
```

```cpp
#include <hip/hip_runtime.h>
#include <rccl/rccl.h>
#include <cstdio>

#define HIP_CHECK(expr) do { hipError_t _e = (expr); if (_e != hipSuccess) { \
    fprintf(stderr, "HIP error %s at %s:%d\n", hipGetErrorString(_e), __FILE__, __LINE__); \
    return; } } while (0)

#define T_OFS 1000
#define P_MAJ 1465
#define P_MIN 768
#define P_PART 1125
#define T_PART 300
#define T_LAND 600
#define T_LANDX 900
#define T_OWN 200
#define T_PL 400
#define W_DELAY 500
#define T_EST 300
#define LAG_INC 400
#define LAG_DEC 100

namespace {

constexpr int NR = 8;
constexpr int NN = 2048;
constexpr int KS = 256;
constexpr int MS = 256;
constexpr int SPL = 1;
constexpr int NWG = 256;
constexpr int NTHREADS = 1024;

constexpr unsigned E_MAJ = 5 * 1024 + SPL * 256;
constexpr unsigned E_MIN = 2 * 1024 + (4 - SPL) * 256;
constexpr unsigned R_MAJ = 160 * E_MAJ, R_MIN = 160 * E_MIN;
constexpr size_t P_OFF = 5ull << 20;
constexpr size_t BUF_BYTES = 7ull << 20;
static_assert(4 * (size_t)(R_MAJ + R_MIN) <= P_OFF, "layout");
constexpr size_t C_OFF = 2 * BUF_BYTES;
constexpr size_t T_OFF = C_OFF + NWG * 4;
constexpr size_t WS_NEED = T_OFF + 128;
constexpr unsigned long long DEADLINE = 150000000ull;
constexpr unsigned TAG16 = 0x40000000u;
constexpr unsigned PAYLOAD = 0xBFFFBFFFu;
constexpr unsigned TAG32 = 1u;
constexpr unsigned NS(int ns) { return (unsigned)ns * 256u / 10u; }

typedef __attribute__((address_space(3))) volatile int lds_vint;
typedef __attribute__((address_space(3))) int lds_int;
typedef float f32x4 __attribute__((ext_vector_type(4)));
typedef unsigned u32x4 __attribute__((ext_vector_type(4)));
typedef _Float16 f16x8 __attribute__((ext_vector_type(8)));
typedef _Float16 f16x2 __attribute__((ext_vector_type(2)));

struct Params {
    const float* x;
    const float* w;
    float* out;
    char* me;
    char* rel[2][4];
    int rank;
};

__device__ __forceinline__ void st16_sys(char* p, u32x4 v) {
    asm volatile("global_store_dwordx4 %0, %1, off sc0 sc1\n\ts_nop 1" :: "v"(p), "v"(v) : "memory");
}
__device__ __forceinline__ void clean_vm() { __builtin_amdgcn_s_waitcnt(0x0F70); }

__device__ __forceinline__ unsigned pack2(float a, float b) {
    f16x2 h = {(_Float16)a, (_Float16)b};
    return __builtin_bit_cast(unsigned, h);
}
__device__ __forceinline__ unsigned or4(u32x4 v) { return (v[0] | v[1]) | (v[2] | v[3]); }

__device__ __forceinline__ unsigned wave_max_u(unsigned v) {
#define DPP_MAX(ctrl, rmask) do { const unsigned t_ = (unsigned)__builtin_amdgcn_update_dpp(0, (int)v, ctrl, rmask, 0xf, false); v = v > t_ ? v : t_; } while (0)
    DPP_MAX(0x111, 0xf); DPP_MAX(0x112, 0xf); DPP_MAX(0x114, 0xf); DPP_MAX(0x118, 0xf);
    DPP_MAX(0x142, 0xa); DPP_MAX(0x143, 0xc);
    return (unsigned)__builtin_amdgcn_readlane((int)v, 63);
}
template <int N> __device__ __forceinline__ int code_of(const f32x4* v) {
    unsigned am = 0;
#pragma unroll
    for (int j = 0; j < N; ++j) {
        const u32x4 u = __builtin_bit_cast(u32x4, v[j]) & 0x7fffffffu;
#pragma unroll
        for (int f = 0; f < 4; ++f) am = u[f] > am ? u[f] : am;
    }
    const int e = (int)(wave_max_u(am) >> 23);
    return e == 255 ? 255 : ((e + 1) | 3) - 1;
}
__device__ __forceinline__ int code_from(u32x4 v, int lane) {
    const int nib = (int)(((v[0] >> 14) & 1u) | ((v[1] >> 13) & 2u) | ((v[2] >> 12) & 4u) | ((v[3] >> 11) & 8u));
    const int oth = __shfl_xor(nib, 1);
    return (lane & 1) ? (nib << 4) | oth : (oth << 4) | nib;
}

struct Spin {
    unsigned long long t0; unsigned it; unsigned call; unsigned* tmo; lds_vint* dead;
    __device__ __forceinline__ bool give_up() {
        __hip_atomic_store(tmo, call, __ATOMIC_RELAXED, __HIP_MEMORY_SCOPE_AGENT);
        *dead = 1;
        return true;
    }
    __device__ __forceinline__ bool fail_lds() {
        __builtin_amdgcn_s_sleep(1);
        asm volatile("" ::: "memory");
        if (__builtin_amdgcn_readfirstlane(*dead) != 0) return true;
        if ((++it & 1023u) == 0u && __builtin_amdgcn_s_memrealtime() - t0 > DEADLINE) return give_up();
        return false;
    }
    __device__ __forceinline__ bool fail() {
        __builtin_amdgcn_s_sleep(2);
        asm volatile("" ::: "memory");
        if ((++it & 63u) != 0u) return false;
        if (__builtin_amdgcn_readfirstlane(*dead)) return true;
        if ((unsigned)__builtin_amdgcn_readfirstlane((int)__hip_atomic_load(tmo, __ATOMIC_RELAXED, __HIP_MEMORY_SCOPE_AGENT)) == call ||
            __builtin_amdgcn_s_memrealtime() - t0 > DEADLINE) return give_up();
        return false;
    }
};

constexpr int L_RED = 0;
constexpr int L_MISC = 131072;
constexpr int L_BYTES = L_MISC + 128;

__global__ void __launch_bounds__(NTHREADS, 1) fused_kernel(Params p) {
    __shared__ __attribute__((aligned(16))) char smem[L_BYTES];
    lds_vint* s_misc = (lds_vint*)(smem + L_MISC);

    const int tid = threadIdx.x, lane = tid & 63;
    const int wave = __builtin_amdgcn_readfirstlane(tid >> 6);
    const int b = blockIdx.x;
    const int rank = p.rank, grp = rank >> 2, gi = rank & 3;
    char* me = p.me;
    const float* px = p.x;
    asm volatile("" :: "s"(px));
    const int j = b >> 3;
    const int c8 = j < 16 ? (j & 7) : (b & 7);
    unsigned* cnt = (unsigned*)(me + C_OFF) + b;
    const unsigned long long tstart = __builtin_amdgcn_s_memrealtime();
#define NOW() ((unsigned)(__builtin_amdgcn_s_memrealtime() - tstart))
#define UNTIL(t_) while (NOW() < ((t_) >> 8)) __builtin_amdgcn_s_sleep(1)

    if (__builtin_expect(j < 16, 1)) {
        const int ks = ((b & 7) - c8) & 7, h = j >> 3;
        const bool split = ks == 5;
        if (wave > 8 || (wave > 4 && !split)) return;
        if (wave == 4) {
            f32x4 v[16];
            UNTIL(NS(W_DELAY));
            const float* src = p.w + (size_t)(32 * ks + 8 * (lane >> 4)) * NN + 64 * (c8 + 8 * h) + 4 * (lane & 15);
#pragma unroll
            for (int q = 0; q < 16; ++q) v[q] = *(const f32x4*)(src + (size_t)(q & 7) * NN + (q >> 3) * 1024);
            const unsigned call = (unsigned)__builtin_amdgcn_readfirstlane((int)__hip_atomic_load(cnt, __ATOMIC_RELAXED, __HIP_MEMORY_SCOPE_AGENT)) + 1u;
            const unsigned tag = ((((call - 1u) >> 1) + 1u) & 1u) ? TAG16 : 0u;
            const int cd = code_of<16>(v);
            const int sh = 126 - cd;
            char* dl = smem + L_RED + ((lane & 15) >> 2) * 1024 + ((lane >> 4) * 16 + 4 * (lane & 3)) * 16;
#pragma unroll
            for (int uh = 0; uh < 2; ++uh)
#pragma unroll
                for (int c = 0; c < 4; ++c) {
                    u32x4 fr;
#pragma unroll
                    for (int q = 0; q < 4; ++q)
                        fr[q] = (cd == 255 ? 0u : pack2(__builtin_ldexpf(v[uh * 8 + 2 * q][c], sh), __builtin_ldexpf(v[uh * 8 + 2 * q + 1][c], sh))) | tag |
                                ((((unsigned)cd >> (4 * (c & 1) + q)) & 1u) << 14);
                    *(u32x4*)(dl + uh * 4096 + c * 16) = fr;
                }
            asm volatile("s_waitcnt lgkmcnt(0)" ::: "memory");
            __builtin_amdgcn_s_barrier();
            if (lane == 0) __hip_atomic_store(cnt, call, __ATOMIC_RELAXED, __HIP_MEMORY_SCOPE_AGENT);
            return;
        }
        const bool own = ks < 5 || (split && wave < 4);
        const bool act = !split || (((lane >> 4) < SPL) == own);
        const int dgrp = own ? grp : grp ^ 1;
        const int dw = wave < 4 ? wave : wave - 5;
        const int dl = (gi + 1 + dw) & 3;
        f32x4 v[4];
        {
            const int row0 = (c8 < 4 ? MS * ((dgrp << 2) | dl) + 64 * c8 : MS * (((dgrp ^ 1) << 2) | (c8 - 4)) + 64 * dl) + 32 * h;
            const float* xs = px + (size_t)(row0 + (lane & 15)) * KS + 32 * ks + 8 * (lane >> 4);
#pragma unroll
            for (int a = 0; a < 2; ++a) {
                v[2 * a] = *(const f32x4*)(xs + (size_t)a * 16 * KS);
                v[2 * a + 1] = *(const f32x4*)(xs + (size_t)a * 16 * KS + 4);
            }
        }
        unsigned call = __hip_atomic_load(cnt, __ATOMIC_RELAXED, __HIP_MEMORY_SCOPE_AGENT) + 1u;
        char* dd = p.rel[own ? 0 : 1][dw];
        call = (unsigned)__builtin_amdgcn_readfirstlane((int)call);
        const unsigned tag = ((((call - 1u) >> 1) + 1u) & 1u) ? TAG16 : 0u;
        const unsigned esz = own ? E_MAJ : E_MIN;
        dd += (size_t)(call & 1u) * BUF_BYTES + lane * 16 +
              (own ? gi * R_MAJ + ks * 1024 : 4 * R_MAJ + gi * R_MIN + (split ? 2048 - SPL * 256 : (ks - 6) * 1024));
        const int cd = code_of<4>(v);
        const int sh = 126 - cd;
        const unsigned nib = (unsigned)cd >> (4 * (lane & 1));
        const u32x4 xtag = {tag | ((nib & 1u) << 14), tag | ((nib & 2u) << 13), tag | ((nib & 4u) << 12), tag | ((nib & 8u) << 11)};
        u32x4 xf[2];
#pragma unroll
        for (int a = 0; a < 2; ++a) {
            xf[a][0] = pack2(__builtin_ldexpf(v[2 * a][0], sh), __builtin_ldexpf(v[2 * a][1], sh));
            xf[a][1] = pack2(__builtin_ldexpf(v[2 * a][2], sh), __builtin_ldexpf(v[2 * a][3], sh));
            xf[a][2] = pack2(__builtin_ldexpf(v[2 * a + 1][0], sh), __builtin_ldexpf(v[2 * a + 1][1], sh));
            xf[a][3] = pack2(__builtin_ldexpf(v[2 * a + 1][2], sh), __builtin_ldexpf(v[2 * a + 1][3], sh));
            if (cd == 255) xf[a] = u32x4{0u, 0u, 0u, 0u};
            xf[a] |= xtag;
        }
        const unsigned dq = own ? NS(P_MAJ) / 16 : NS(P_MIN) / 16;
        unsigned t00 = min(NOW(), 1u << 22) << 8;
        t00 = t00 > NS(T_OFS) ? t00 : NS(T_OFS);
        const unsigned xs0 = (unsigned)(((c8 ^ 4) * 2 + h) * 2);
        const bool early = split && own && c8 < 4;
        if (early) { __builtin_amdgcn_s_barrier(); asm volatile("" ::: "memory"); }
        const unsigned tx = t00 + dq * xs0;
        UNTIL(tx);
        if (act) st16_sys(dd + xs0 * esz, xf[0]);
        UNTIL(tx + dq);
        if (act) st16_sys(dd + (xs0 + 1) * esz, xf[1]);
        if (!early) { __builtin_amdgcn_s_barrier(); asm volatile("" ::: "memory"); }
        char* wd = dd + (unsigned)(32 + c8 * 2 + h) * esz;
        const char* ls = smem + L_RED + lane * 16;
        unsigned due = t00 + dq * (unsigned)(32 + c8 * 2 + h);
#pragma unroll 1
        for (int s = 0; s < 8; ++s) {
            const u32x4 fr = *(const u32x4*)(ls + (s & 1) * 4096 + (s >> 1) * 1024);
            UNTIL(due);
            due += 16 * dq;
            if (act) st16_sys(wd + (unsigned)s * 16u * esz, fr);
        }
        return;
    }

    const int jc = j - 16, u0 = jc & 1, kind = 1 - ((jc >> 1) & 1), rq = jc >> 2;
    unsigned call = __hip_atomic_load(cnt, __ATOMIC_RELAXED, __HIP_MEMORY_SCOPE_AGENT) + 1u;
    if (wave == 9 && lane < 17) s_misc[lane] = 0;
    unsigned pfa = 0;
    if (wave == 8 && jc == 2) {
        const int d = lane & 3, g = (lane >> 2) & 1, e = (lane >> 4) & 1;
        const char* rb = (g == 0 ? (d == 0 ? p.rel[0][0] : d == 1 ? p.rel[0][1] : d == 2 ? p.rel[0][2] : p.rel[0][3])
                                 : (d == 0 ? p.rel[1][0] : d == 1 ? p.rel[1][1] : d == 2 ? p.rel[1][2] : p.rel[1][3])) +
                         (size_t)((lane >> 3) & 1) * BUF_BYTES;
        const unsigned sub = (unsigned)c8 * 128u;
        const unsigned o = lane < 32 ? (g == 0 ? gi * R_MAJ + (e ? R_MAJ - 16384 : 0) : 4 * R_MAJ + gi * R_MIN + (e ? R_MIN - 16384 : 0))
                                     : (unsigned)P_OFF + gi * 524288 + (e ? 524288 - 16384 : 0);
        const int li = (lane & 3) | ((lane >> 1) & 12);
        const char* a = (lane < 32 || g == 1) ? rb + o : (li < 12 ? me + (size_t)li * (5u << 18) : (const char*)p.out + (li - 12) * 524288);
        pfa = __builtin_nontemporal_load((const unsigned*)(a + sub + 8192));
    }
    call = (unsigned)__builtin_amdgcn_readfirstlane((int)call);
    __syncthreads();
    unsigned tb = (min(NOW(), 1u << 22) << 8) + NS(T_EST);
    tb = tb > NS(T_OFS) ? tb : NS(T_OFS);
    asm volatile("" :: "v"(pfa));
    if (tid == 0) __hip_atomic_store(cnt, call, __ATOMIC_RELAXED, __HIP_MEMORY_SCOPE_AGENT);
    const bool ph = ((((call - 1u) >> 1) + 1u) & 1u) != 0u;
    const size_t boff = (size_t)(call & 1u) * BUF_BYTES;
    Spin sp; sp.t0 = tstart; sp.it = 0; sp.call = call; sp.tmo = (unsigned*)(me + T_OFF); sp.dead = s_misc + 16;
    bool dead = false;
#define LDS_WAIT(idx, n) while (!dead && __builtin_amdgcn_readfirstlane(s_misc[idx]) < (n)) { if (sp.fail_lds()) dead = true; }
#define LDS_SIGNAL(idx) do { if (lane == 0) __hip_atomic_fetch_add((lds_int*)(s_misc + (idx)), 1, __ATOMIC_RELAXED, __HIP_MEMORY_SCOPE_WORKGROUP); } while (0)

    if (wave >= 8) {
        clean_vm();
        const int a = wave & 3;
        const unsigned tag = ph ? TAG32 : 0u;
        char* pp = (kind ? p.rel[1][(rq - gi - 1) & 3] : me) + boff + P_OFF + (size_t)(kind ? gi : rq) * 524288 + (c8 * 2 + u0) * 4096 + a * 1024 + lane * 16;
        float* op = p.out + (size_t)(64 * rq + 16 * a + (lane & 15)) * NN + 64 * (c8 + 8 * u0) + 4 * (lane >> 4);
#pragma unroll 1
        for (int s = (wave >> 2) & 1; s < 8; s += 2) {
            LDS_WAIT(s, 8);
            asm volatile("" ::: "memory");
            const char* rp = smem + L_RED + (s & 3) * 32768 + a * 1024 + lane * 16;
            f32x4 sum = *(const f32x4*)rp;
#pragma unroll
            for (int w = 1; w < 8; ++w) sum += *(const f32x4*)(rp + w * 4096);
            asm volatile("" : "+v"(sum) :: "memory");
            LDS_SIGNAL(8 + s);
            const unsigned tp = tb + NS(10 * P_MIN + T_PART) + NS(P_PART) / 16 * (unsigned)(s * 16 + c8 * 2 + u0);
            if (kind) {
                UNTIL(tp);
                if (!dead) st16_sys(pp + s * 65536, (__builtin_bit_cast(u32x4, sum) & ~TAG32) | tag);
            } else {
                UNTIL(tp + NS(T_PL));
                u32x4 pv = {0u, 0u, 0u, 0u};
                while (!dead) {
                    asm volatile("global_load_dwordx4 %0, %1, off sc0 sc1\n\ts_waitcnt vmcnt(0)" : "=&v"(pv) : "v"(pp + s * 65536) : "memory");
                    pv ^= tag;
                    if (__ballot((or4(pv) & TAG32) != 0u) == 0ull) break;
                    if (sp.fail()) dead = true;
                }
                const f32x4 y = sum + __builtin_bit_cast(f32x4, pv);
                f32x4 o;
#pragma unroll
                for (int r = 0; r < 4; ++r) {
                    const float c = fminf(fmaxf(y[r], -60.f), 60.f);
                    o[r] = y[r] / (1.f + __expf(-c));
                }
                if (dead) o = f32x4{__builtin_nanf(""), __builtin_nanf(""), __builtin_nanf(""), __builtin_nanf("")};
                st16_sys((char*)(op + (s & 1) * 1024 + (s >> 1) * 16), __builtin_bit_cast(u32x4, o));
            }
        }
        return;
    }

    const int t0 = wave * 4;
    const unsigned tag = ph ? TAG16 : 0u;
    const bool fast = wave == 5 || wave == 6;
    const unsigned dq = fast ? NS(P_MIN) / 16 : NS(P_MAJ) / 16;
    const bool lmaj = wave < 5 || (wave == 7 && (lane >> 4) < SPL);
    const unsigned esz = lmaj ? E_MAJ : E_MIN;
    unsigned fo[4];
#pragma unroll
    for (int tt = 0; tt < 4; ++tt) {
        const int t = t0 + tt;
        fo[tt] = lane * 16 + (wave < 5 ? (t / 5) * R_MAJ + (t % 5) * 1024
                              : wave < 7 ? 4 * R_MAJ + ((t - 20) >> 1) * R_MIN + ((t - 20) & 1) * 1024
                              : lmaj ? tt * R_MAJ + 5120 : 4 * R_MAJ + tt * R_MIN + 2048 - SPL * 256);
    }
    __amdgpu_buffer_rsrc_t xr = __builtin_amdgcn_make_buffer_rsrc((void*)(me + boff), 0, 5 << 20, 0x00020000);
    const unsigned xs0 = (unsigned)((kind ? 0 : 16) + 4 * rq) * esz;
    UNTIL(tb + NS(T_LANDX) + dq * (unsigned)((kind ? 0 : 16) + 4 * rq + 3));
    while (!dead) {
        unsigned v = tag;
        if ((lane & 12) == 12 && ((lane >> 4) == 3 || (lane >> 4) == SPL - 1)) {
            const int tt = lane & 3;
            v = __builtin_amdgcn_raw_buffer_load_b32(xr, (tt == 0 ? fo[0] : tt == 1 ? fo[1] : tt == 2 ? fo[2] : fo[3]) + xs0 + 3 * esz, 0, 17);
        }
        if (__ballot(((v ^ tag) & TAG16) != 0u) == 0ull) break;
        __builtin_amdgcn_s_sleep(12);
        if (sp.fail()) dead = true;
    }
    u32x4 af[4][4];
    while (!dead) {
        unsigned m = 0u;
#pragma unroll
        for (int tt = 0; tt < 4; ++tt)
#pragma unroll
            for (int a = 0; a < 4; ++a)
                af[tt][a] = __builtin_amdgcn_raw_buffer_load_b128(xr, fo[tt] + xs0 + a * esz, 0, 17);
#pragma unroll
        for (int tt = 0; tt < 4; ++tt)
#pragma unroll
            for (int a = 0; a < 4; ++a) { af[tt][a] ^= tag; m |= or4(af[tt][a]); }
        if (__ballot((m & TAG16) != 0u) == 0ull) break;
        __builtin_amdgcn_s_sleep(12);
        if (sp.fail()) dead = true;
    }
    clean_vm();
    int cx[4][2], ex[2] = {0, 0};
#pragma unroll
    for (int tt = 0; tt < 4; ++tt) {
        cx[tt][0] = code_from(af[tt][0], lane); cx[tt][1] = code_from(af[tt][2], lane);
#pragma unroll
        for (int a = 0; a < 4; ++a) af[tt][a] &= PAYLOAD;
    }
    const char* wq = me + boff;
#pragma unroll
    for (int tt = 0; tt < 4; ++tt) fo[tt] += (unsigned)(32 + c8 * 2 + u0) * esz;
    unsigned due = tb + NS(T_LAND) + (fast ? NS(300) : (kind ? 0u : NS(T_OWN))) + dq * (unsigned)(32 + c8 * 2 + u0);
    unsigned lag = 0;
#pragma unroll 1
    for (int s = 0; s < 8; ++s) {
        const unsigned g = due >> 8;
        due += 16 * dq;
        while (NOW() < g + lag) __builtin_amdgcn_s_sleep(1);
        lag = lag > LAG_DEC / 10 ? lag - LAG_DEC / 10 : 0;
        u32x4 cur[4];
        for (;;) {
            const unsigned t_try = NOW();
            asm volatile("s_nop 4\n\tglobal_load_dwordx4 %0, %4, %8 sc0 sc1\n\tglobal_load_dwordx4 %1, %5, %8 sc0 sc1\n\t"
                         "global_load_dwordx4 %2, %6, %8 sc0 sc1\n\tglobal_load_dwordx4 %3, %7, %8 sc0 sc1\n\t"
                         "s_waitcnt vmcnt(0)"
                         : "=&v"(cur[0]), "=&v"(cur[1]), "=&v"(cur[2]), "=&v"(cur[3])
                         : "v"(fo[0]), "v"(fo[1]), "v"(fo[2]), "v"(fo[3]), "s"(wq) : "memory");
#pragma unroll
            for (int tt = 0; tt < 4; ++tt) cur[tt] ^= tag;
            if (dead || __ballot(((or4(cur[0]) | or4(cur[1]) | or4(cur[2]) | or4(cur[3])) & TAG16) != 0u) == 0ull) break;
            lag = t_try + LAG_INC / 10 - g;
            __builtin_amdgcn_s_sleep(6);
            if (sp.fail()) { dead = true; break; }
        }
#pragma unroll
        for (int tt = 0; tt < 4; ++tt) fo[tt] += 16u * esz;
        if (s == 0) {
            bool bad = false, same = true;
#pragma unroll
            for (int tt = 0; tt < 4; ++tt) {
                const int cw = code_from(cur[tt], lane);
                bad |= cw == 255 || cx[tt][0] == 255 || cx[tt][1] == 255;
                cx[tt][0] += cw; cx[tt][1] += cw;
            }
            ex[0] = __builtin_amdgcn_readfirstlane(cx[0][0]); ex[1] = __builtin_amdgcn_readfirstlane(cx[0][1]);
#pragma unroll
            for (int tt = 0; tt < 4; ++tt) same &= cx[tt][0] == ex[0] && cx[tt][1] == ex[1];
            if (__ballot(bad || !same) != 0ull) {
                const bool poison = __ballot(bad) != 0ull;
#pragma unroll
                for (int hh = 0; hh < 2; ++hh) {
                    ex[hh] = (int)wave_max_u((unsigned)max(max(cx[0][hh], cx[1][hh]), max(cx[2][hh], cx[3][hh])));
#pragma unroll
                    for (int tt = 0; tt < 4; ++tt) {
                        const _Float16 m = poison ? (_Float16)__builtin_nanf("") : (_Float16)__builtin_ldexpf(1.f, cx[tt][hh] - ex[hh]);
#pragma unroll
                        for (int a = 0; a < 2; ++a) {
                            f16x8 t = __builtin_bit_cast(f16x8, af[tt][2 * hh + a]);
                            t *= m;
                            af[tt][2 * hh + a] = __builtin_bit_cast(u32x4, t);
                        }
                    }
                }
            }
            ex[0] -= 252; ex[1] -= 252;
        }
#pragma unroll
        for (int tt = 0; tt < 4; ++tt) cur[tt] &= PAYLOAD;
        f32x4 acc[4];
#pragma unroll
        for (int a = 0; a < 4; ++a) acc[a] = f32x4{0.f, 0.f, 0.f, 0.f};
#pragma unroll
        for (int tt = 0; tt < 4; ++tt)
#pragma unroll
            for (int a = 0; a < 4; ++a)
                acc[a] = __builtin_amdgcn_mfma_f32_16x16x32_f16(__builtin_bit_cast(f16x8, cur[tt]), __builtin_bit_cast(f16x8, af[tt][a]), acc[a], 0, 0, 0);
        if (s >= 4) LDS_WAIT(8 + s - 4, 4);
        asm volatile("" ::: "memory");
        char* rp = smem + L_RED + (s & 3) * 32768 + wave * 4096 + lane * 16;
#pragma unroll
        for (int a = 0; a < 4; ++a) {
            const int e = ex[a >> 1];
            *(f32x4*)(rp + a * 1024) = f32x4{__builtin_ldexpf(acc[a][0], e), __builtin_ldexpf(acc[a][1], e),
                                             __builtin_ldexpf(acc[a][2], e), __builtin_ldexpf(acc[a][3], e)};
        }
        asm volatile("" ::: "memory");
        LDS_SIGNAL(s);
    }
}

}

extern "C" void kernel_launch(int rank, int nranks, ncclComm_t comm,
                              hipStream_t stream,
                              void* const* d_in, const int* in_sizes, int n_in,
                              void* d_out, int out_size,
                              const int* in_dtypes, int out_dtype,
                              void* const* d_out_all,
                              void* const* d_ws_all, size_t ws_bytes) {
    if (nranks != NR || (unsigned)rank >= (unsigned)NR || ws_bytes < WS_NEED || n_in < 2) {
        fprintf(stderr, "unexpected configuration\n");
        return;
    }
    Params p;
    p.x = (const float*)d_in[0];
    p.w = (const float*)d_in[1];
    p.out = (float*)d_out;
    p.me = (char*)d_ws_all[rank];
    for (int g = 0; g < 2; ++g)
        for (int d = 0; d < 4; ++d) p.rel[g][d] = (char*)d_ws_all[(((rank >> 2) ^ g) << 2) | ((rank + 1 + d) & 3)];
    p.rank = rank;
    hipLaunchKernelGGL(fused_kernel, dim3(NWG), dim3(NTHREADS), 0, stream, p);
    HIP_CHECK(hipGetLastError());
}
```

```cpp
#include <hip/hip_runtime.h>
#include <rccl/rccl.h>
#include <cstdio>

#define HIP_CHECK(expr) do { hipError_t _e = (expr); if (_e != hipSuccess) { \
    fprintf(stderr, "HIP error %s at %s:%d\n", hipGetErrorString(_e), __FILE__, __LINE__); \
    return; } } while (0)

#define T_OFS 1000
#define P_MAJ 1465
#define P_MIN 768
#define P_PART 1125
#define T_PART 300
#define T_LAND 600
#define T_LANDX 900
#define T_OWN 200
#define T_PL 400
#define W_DELAY 500
#define T_EST 300
#define T_READ 3000
#define T_COM 100
#define LAG_INC 400
#define LAG_DEC 100

namespace {

constexpr int NR = 8;
constexpr int NN = 2048;
constexpr int KS = 256;
constexpr int MS = 256;
constexpr int SPL = 1;
constexpr int NWG = 256;
constexpr int NTHREADS = 1024;

constexpr unsigned E_MAJ = 5 * 1024 + SPL * 256;
constexpr unsigned E_MIN = 2 * 1024 + (4 - SPL) * 256;
constexpr unsigned R_MAJ = 160 * E_MAJ, R_MIN = 160 * E_MIN;
constexpr size_t P_OFF = 5ull << 20;
constexpr size_t BUF_BYTES = 7ull << 20;
static_assert(4 * (size_t)(R_MAJ + R_MIN) <= P_OFF, "layout");
constexpr size_t C_OFF = 2 * BUF_BYTES;
constexpr size_t T_OFF = C_OFF + NWG * 4;
constexpr size_t R_OFF = T_OFF + 128 - (T_OFF & 127) + 128;
constexpr size_t WS_NEED = R_OFF + 256;
constexpr unsigned long long DEADLINE = 150000000ull;
constexpr unsigned TAG16 = 0x40000000u;
constexpr unsigned PAYLOAD = 0xBFFFBFFFu;
constexpr unsigned TAG32 = 1u;
constexpr unsigned NS(int ns) { return (unsigned)ns * 256u / 10u; }

typedef __attribute__((address_space(3))) volatile int lds_vint;
typedef __attribute__((address_space(3))) int lds_int;
typedef float f32x4 __attribute__((ext_vector_type(4)));
typedef unsigned u32x4 __attribute__((ext_vector_type(4)));
typedef _Float16 f16x8 __attribute__((ext_vector_type(8)));
typedef _Float16 f16x2 __attribute__((ext_vector_type(2)));

struct Params {
    const float* x;
    const float* w;
    float* out;
    char* me;
    char* rel[2][4];
    int rank;
};

__device__ __forceinline__ void st16_sys(char* p, u32x4 v) {
    asm volatile("global_store_dwordx4 %0, %1, off sc0 sc1\n\ts_nop 1" :: "v"(p), "v"(v) : "memory");
}
__device__ __forceinline__ void clean_vm() { __builtin_amdgcn_s_waitcnt(0x0F70); }

__device__ __forceinline__ unsigned pack2(float a, float b) {
    f16x2 h = {(_Float16)a, (_Float16)b};
    return __builtin_bit_cast(unsigned, h);
}
__device__ __forceinline__ unsigned or4(u32x4 v) { return (v[0] | v[1]) | (v[2] | v[3]); }

__device__ __forceinline__ unsigned wave_max_u(unsigned v) {
#define DPP_MAX(ctrl, rmask) do { const unsigned t_ = (unsigned)__builtin_amdgcn_update_dpp(0, (int)v, ctrl, rmask, 0xf, false); v = v > t_ ? v : t_; } while (0)
    DPP_MAX(0x111, 0xf); DPP_MAX(0x112, 0xf); DPP_MAX(0x114, 0xf); DPP_MAX(0x118, 0xf);
    DPP_MAX(0x142, 0xa); DPP_MAX(0x143, 0xc);
    return (unsigned)__builtin_amdgcn_readlane((int)v, 63);
}
template <int N> __device__ __forceinline__ int code_of(const f32x4* v) {
    unsigned am = 0;
#pragma unroll
    for (int j = 0; j < N; ++j) {
        const u32x4 u = __builtin_bit_cast(u32x4, v[j]) & 0x7fffffffu;
#pragma unroll
        for (int f = 0; f < 4; ++f) am = u[f] > am ? u[f] : am;
    }
    const int e = (int)(wave_max_u(am) >> 23);
    return e == 255 ? 255 : ((e + 1) | 3) - 1;
}
__device__ __forceinline__ int code_from(u32x4 v, int lane) {
    const int nib = (int)(((v[0] >> 14) & 1u) | ((v[1] >> 13) & 2u) | ((v[2] >> 12) & 4u) | ((v[3] >> 11) & 8u));
    const int oth = __shfl_xor(nib, 1);
    return (lane & 1) ? (nib << 4) | oth : (oth << 4) | nib;
}

__device__ __forceinline__ unsigned mean_ready(const char* me, unsigned call, unsigned long long tstart, int lane, unsigned fallback) {
    const unsigned long long e = __hip_atomic_load((const unsigned long long*)(me + R_OFF) + (lane & 31), __ATOMIC_RELAXED, __HIP_MEMORY_SCOPE_AGENT);
    const bool ok = (unsigned)(e >> 32) == call && lane < 32;
    int d = ok ? (int)((unsigned)e - (unsigned)tstart) : 0;
    d = d < -4096 ? -4096 : (d > 65536 ? 65536 : d);
#pragma unroll
    for (int o = 16; o >= 1; o >>= 1) d += __shfl_xor(d, o);
    const int n = __popcll(__ballot(ok));
    d = __builtin_amdgcn_readfirstlane(d);
    return n < 8 || d < 0 ? fallback : (unsigned)(d * 256 / n);
}

struct Spin {
    unsigned long long t0; unsigned it; unsigned call; unsigned* tmo; lds_vint* dead;
    __device__ __forceinline__ bool give_up() {
        __hip_atomic_store(tmo, call, __ATOMIC_RELAXED, __HIP_MEMORY_SCOPE_AGENT);
        *dead = 1;
        return true;
    }
    __device__ __forceinline__ bool fail_lds() {
        __builtin_amdgcn_s_sleep(1);
        asm volatile("" ::: "memory");
        if (__builtin_amdgcn_readfirstlane(*dead) != 0) return true;
        if ((++it & 1023u) == 0u && __builtin_amdgcn_s_memrealtime() - t0 > DEADLINE) return give_up();
        return false;
    }
    __device__ __forceinline__ bool fail() {
        __builtin_amdgcn_s_sleep(2);
        asm volatile("" ::: "memory");
        if ((++it & 63u) != 0u) return false;
        if (__builtin_amdgcn_readfirstlane(*dead)) return true;
        if ((unsigned)__builtin_amdgcn_readfirstlane((int)__hip_atomic_load(tmo, __ATOMIC_RELAXED, __HIP_MEMORY_SCOPE_AGENT)) == call ||
            __builtin_amdgcn_s_memrealtime() - t0 > DEADLINE) return give_up();
        return false;
    }
};

constexpr int L_RED = 0;
constexpr int L_MISC = 131072;
constexpr int L_BYTES = L_MISC + 128;

__global__ void __launch_bounds__(NTHREADS, 1) fused_kernel(Params p) {
    __shared__ __attribute__((aligned(16))) char smem[L_BYTES];
    lds_vint* s_misc = (lds_vint*)(smem + L_MISC);

    const int tid = threadIdx.x, lane = tid & 63;
    const int wave = __builtin_amdgcn_readfirstlane(tid >> 6);
    const int b = blockIdx.x;
    const int rank = p.rank, grp = rank >> 2, gi = rank & 3;
    char* me = p.me;
    const float* px = p.x;
    asm volatile("" :: "s"(px));
    const int j = b >> 3;
    const int c8 = j < 16 ? (j & 7) : (b & 7);
    unsigned* cnt = (unsigned*)(me + C_OFF) + b;
    const unsigned long long tstart = __builtin_amdgcn_s_memrealtime();
#define NOW() ((unsigned)(__builtin_amdgcn_s_memrealtime() - tstart))
#define UNTIL(t_) while (NOW() < ((t_) >> 8)) __builtin_amdgcn_s_sleep(1)

    if (__builtin_expect(j < 16, 1)) {
        const int ks = ((b & 7) - c8) & 7, h = j >> 3;
        const bool split = ks == 5;
        if (wave > 8 || (wave > 4 && !split)) return;
        if (wave == 4) {
            f32x4 v[16];
            if (ks < 5) UNTIL(NS(W_DELAY));
            const float* src = p.w + (size_t)(32 * ks + 8 * (lane >> 4)) * NN + 64 * (c8 + 8 * h) + 4 * (lane & 15);
#pragma unroll
            for (int q = 0; q < 16; ++q) v[q] = *(const f32x4*)(src + (size_t)(q & 7) * NN + (q >> 3) * 1024);
            const unsigned call = (unsigned)__builtin_amdgcn_readfirstlane((int)__hip_atomic_load(cnt, __ATOMIC_RELAXED, __HIP_MEMORY_SCOPE_AGENT)) + 1u;
            const unsigned tag = ((((call - 1u) >> 1) + 1u) & 1u) ? TAG16 : 0u;
            const int cd = code_of<16>(v);
            const float sc = __builtin_ldexpf(1.f, 126 - cd);
            const unsigned keep = cd == 255 ? 0u : ~0u;
            char* dl = smem + L_RED + ((lane & 15) >> 2) * 1024 + ((lane >> 4) * 16 + 4 * (lane & 3)) * 16;
#pragma unroll
            for (int q = 0; q < 16; ++q) v[q] *= sc;
#pragma unroll
            for (int uh = 0; uh < 2; ++uh)
#pragma unroll
                for (int c = 0; c < 4; ++c) {
                    u32x4 fr;
#pragma unroll
                    for (int q = 0; q < 4; ++q)
                        fr[q] = (pack2(v[uh * 8 + 2 * q][c], v[uh * 8 + 2 * q + 1][c]) & keep) | (tag | ((((unsigned)cd >> (4 * (c & 1) + q)) & 1u) << 14));
                    *(u32x4*)(dl + uh * 4096 + c * 16) = fr;
                }
            if (ks < 5) {
                UNTIL(NS(T_READ));
                const unsigned m = mean_ready(me, call, tstart, lane, 0u);
                if (lane == 0) s_misc[0] = m ? (int)((unsigned)tstart + (m >> 8)) | 1 : 0;
            }
            asm volatile("s_waitcnt lgkmcnt(0)" ::: "memory");
            __builtin_amdgcn_s_barrier();
            if (lane == 0) __hip_atomic_store(cnt, call, __ATOMIC_RELAXED, __HIP_MEMORY_SCOPE_AGENT);
            return;
        }
        const bool own = ks < 5 || (split && wave < 4);
        const bool act = !split || (((lane >> 4) < SPL) == own);
        const int dgrp = own ? grp : grp ^ 1;
        const int dw = wave < 4 ? wave : wave - 5;
        const int dl = (gi + 1 + dw) & 3;
        f32x4 v[4];
        {
            const int row0 = (c8 < 4 ? MS * ((dgrp << 2) | dl) + 64 * c8 : MS * (((dgrp ^ 1) << 2) | (c8 - 4)) + 64 * dl) + 32 * h;
            const float* xs = px + (size_t)(row0 + (lane & 15)) * KS + 32 * ks + 8 * (lane >> 4);
#pragma unroll
            for (int a = 0; a < 2; ++a) {
                v[2 * a] = *(const f32x4*)(xs + (size_t)a * 16 * KS);
                v[2 * a + 1] = *(const f32x4*)(xs + (size_t)a * 16 * KS + 4);
            }
        }
        unsigned call = __hip_atomic_load(cnt, __ATOMIC_RELAXED, __HIP_MEMORY_SCOPE_AGENT) + 1u;
        char* dd = p.rel[own ? 0 : 1][dw];
        call = (unsigned)__builtin_amdgcn_readfirstlane((int)call);
        const unsigned tag = ((((call - 1u) >> 1) + 1u) & 1u) ? TAG16 : 0u;
        const unsigned esz = own ? E_MAJ : E_MIN;
        dd += (size_t)(call & 1u) * BUF_BYTES + lane * 16 +
              (own ? gi * R_MAJ + ks * 1024 : 4 * R_MAJ + gi * R_MIN + (split ? 2048 - SPL * 256 : (ks - 6) * 1024));
        const int cd = code_of<4>(v);
        const int sh = 126 - cd;
        const unsigned nib = (unsigned)cd >> (4 * (lane & 1));
        const u32x4 xtag = {tag | ((nib & 1u) << 14), tag | ((nib & 2u) << 13), tag | ((nib & 4u) << 12), tag | ((nib & 8u) << 11)};
        u32x4 xf[2];
#pragma unroll
        for (int a = 0; a < 2; ++a) {
            xf[a][0] = pack2(__builtin_ldexpf(v[2 * a][0], sh), __builtin_ldexpf(v[2 * a][1], sh));
            xf[a][1] = pack2(__builtin_ldexpf(v[2 * a][2], sh), __builtin_ldexpf(v[2 * a][3], sh));
            xf[a][2] = pack2(__builtin_ldexpf(v[2 * a + 1][0], sh), __builtin_ldexpf(v[2 * a + 1][1], sh));
            xf[a][3] = pack2(__builtin_ldexpf(v[2 * a + 1][2], sh), __builtin_ldexpf(v[2 * a + 1][3], sh));
            if (cd == 255) xf[a] = u32x4{0u, 0u, 0u, 0u};
            xf[a] |= xtag;
        }
        const unsigned dq = own ? NS(P_MAJ) / 16 : NS(P_MIN) / 16;
        unsigned t00 = min(NOW(), 1u << 22) << 8;
        t00 = t00 > NS(T_OFS) ? t00 : NS(T_OFS);
        if (wave == (j & 3) && (j % 5) == 0 && lane == 0)
            __hip_atomic_store((unsigned long long*)(me + R_OFF) + (b & 7) * 4 + j / 5,
                               ((unsigned long long)call << 32) | ((unsigned)tstart + (t00 >> 8)), __ATOMIC_RELAXED, __HIP_MEMORY_SCOPE_AGENT);
        const unsigned xs0 = (unsigned)(((c8 ^ 4) * 2 + h) * 2);
        const bool early = split && own && c8 < 4;
        if (early) { __builtin_amdgcn_s_barrier(); asm volatile("" ::: "memory"); }
        const unsigned tx = t00 + dq * xs0;
        UNTIL(tx);
        if (act) st16_sys(dd + xs0 * esz, xf[0]);
        UNTIL(tx + dq);
        if (act) st16_sys(dd + (xs0 + 1) * esz, xf[1]);
        if (!early) { __builtin_amdgcn_s_barrier(); asm volatile("" ::: "memory"); }
        char* wd = dd + (unsigned)(32 + c8 * 2 + h) * esz;
        const char* ls = smem + L_RED + lane * 16;
        if (ks < 5) {
            const unsigned o = (unsigned)__builtin_amdgcn_readfirstlane(s_misc[0]);
            const int rel = (int)(o - (unsigned)tstart);
            if (o != 0u && rel > 0 && rel < 65536) t00 = ((unsigned)rel << 8) + NS(T_COM);
        }
        if (split && own) {
            UNTIL(NS(T_READ));
            t00 = mean_ready(me, call, tstart, lane, t00) + NS(T_COM);
            clean_vm();
        }
        unsigned due = t00 + dq * (unsigned)(32 + c8 * 2 + h);
#pragma unroll 1
        for (int s = 0; s < 8; ++s) {
            const u32x4 fr = *(const u32x4*)(ls + (s & 1) * 4096 + (s >> 1) * 1024);
            UNTIL(due);
            due += 16 * dq;
            if (act) st16_sys(wd + (unsigned)s * 16u * esz, fr);
        }
        return;
    }

    const int jc = j - 16, u0 = jc & 1, kind = 1 - ((jc >> 1) & 1), rq = jc >> 2;
    unsigned call = __hip_atomic_load(cnt, __ATOMIC_RELAXED, __HIP_MEMORY_SCOPE_AGENT) + 1u;
    if (wave == 9 && lane < 17) s_misc[lane] = 0;
    unsigned pfa = 0;
    if (wave == 8 && jc == 2) {
        const int d = lane & 3, g = (lane >> 2) & 1, e = (lane >> 4) & 1;
        const char* rb = (g == 0 ? (d == 0 ? p.rel[0][0] : d == 1 ? p.rel[0][1] : d == 2 ? p.rel[0][2] : p.rel[0][3])
                                 : (d == 0 ? p.rel[1][0] : d == 1 ? p.rel[1][1] : d == 2 ? p.rel[1][2] : p.rel[1][3])) +
                         (size_t)((lane >> 3) & 1) * BUF_BYTES;
        const unsigned sub = (unsigned)c8 * 128u;
        const unsigned o = lane < 32 ? (g == 0 ? gi * R_MAJ + (e ? R_MAJ - 16384 : 0) : 4 * R_MAJ + gi * R_MIN + (e ? R_MIN - 16384 : 0))
                                     : (unsigned)P_OFF + gi * 524288 + (e ? 524288 - 16384 : 0);
        const int li = (lane & 3) | ((lane >> 1) & 12);
        const char* a = (lane < 32 || g == 1) ? rb + o : (li < 12 ? me + (size_t)li * (5u << 18) : (const char*)p.out + (li - 12) * 524288);
        pfa = __builtin_nontemporal_load((const unsigned*)(a + sub + 8192));
    }
    call = (unsigned)__builtin_amdgcn_readfirstlane((int)call);
    __syncthreads();
    unsigned tb = (min(NOW(), 1u << 22) << 8) + NS(T_EST);
    tb = tb > NS(T_OFS) ? tb : NS(T_OFS);
    asm volatile("" :: "v"(pfa));
    if (tid == 0) __hip_atomic_store(cnt, call, __ATOMIC_RELAXED, __HIP_MEMORY_SCOPE_AGENT);
    const bool ph = ((((call - 1u) >> 1) + 1u) & 1u) != 0u;
    const size_t boff = (size_t)(call & 1u) * BUF_BYTES;
    Spin sp; sp.t0 = tstart; sp.it = 0; sp.call = call; sp.tmo = (unsigned*)(me + T_OFF); sp.dead = s_misc + 16;
    bool dead = false;
#define LDS_WAIT(idx, n) while (!dead && __builtin_amdgcn_readfirstlane(s_misc[idx]) < (n)) { if (sp.fail_lds()) dead = true; }
#define LDS_SIGNAL(idx) do { if (lane == 0) __hip_atomic_fetch_add((lds_int*)(s_misc + (idx)), 1, __ATOMIC_RELAXED, __HIP_MEMORY_SCOPE_WORKGROUP); } while (0)

    if (wave >= 8) {
        clean_vm();
        const int a = wave & 3;
        const unsigned tag = ph ? TAG32 : 0u;
        char* pp = (kind ? p.rel[1][(rq - gi - 1) & 3] : me) + boff + P_OFF + (size_t)(kind ? gi : rq) * 524288 + (c8 * 2 + u0) * 4096 + a * 1024 + lane * 16;
        float* op = p.out + (size_t)(64 * rq + 16 * a + (lane & 15)) * NN + 64 * (c8 + 8 * u0) + 4 * (lane >> 4);
        UNTIL(tb + NS(3000));
        tb = mean_ready(me, call, tstart, lane, tb);
        clean_vm();
#pragma unroll 1
        for (int s = (wave >> 2) & 1; s < 8; s += 2) {
            LDS_WAIT(s, 8);
            asm volatile("" ::: "memory");
            const char* rp = smem + L_RED + (s & 3) * 32768 + a * 1024 + lane * 16;
            f32x4 sum = *(const f32x4*)rp;
#pragma unroll
            for (int w = 1; w < 8; ++w) sum += *(const f32x4*)(rp + w * 4096);
            if (__builtin_amdgcn_readfirstlane(s_misc[16]) != 0) dead = true;
            asm volatile("" : "+v"(sum) :: "memory");
            LDS_SIGNAL(8 + s);
            const unsigned tp = tb + NS(10 * P_MIN + T_PART) + NS(P_PART) / 16 * (unsigned)(s * 16 + c8 * 2 + u0);
            if (kind) {
                UNTIL(tp);
                if (!dead) st16_sys(pp + s * 65536, (__builtin_bit_cast(u32x4, sum) & ~TAG32) | tag);
            } else {
                UNTIL(tp + NS(T_PL));
                u32x4 pv = {0u, 0u, 0u, 0u};
                while (!dead) {
                    asm volatile("global_load_dwordx4 %0, %1, off sc0 sc1\n\ts_waitcnt vmcnt(0)" : "=&v"(pv) : "v"(pp + s * 65536) : "memory");
                    pv ^= tag;
                    if (__ballot((or4(pv) & TAG32) != 0u) == 0ull) break;
                    if (sp.fail()) dead = true;
                }
                const f32x4 y = sum + __builtin_bit_cast(f32x4, pv);
                f32x4 o;
#pragma unroll
                for (int r = 0; r < 4; ++r) {
                    const float c = fminf(fmaxf(y[r], -60.f), 60.f);
                    o[r] = y[r] / (1.f + __expf(-c));
                }
                if (dead) o = f32x4{__builtin_nanf(""), __builtin_nanf(""), __builtin_nanf(""), __builtin_nanf("")};
                st16_sys((char*)(op + (s & 1) * 1024 + (s >> 1) * 16), __builtin_bit_cast(u32x4, o));
            }
        }
        return;
    }

    const int t0 = wave * 4;
    const unsigned tag = ph ? TAG16 : 0u;
    const bool fast = wave == 5 || wave == 6;
    const unsigned dq = fast ? NS(P_MIN) / 16 : NS(P_MAJ) / 16;
    const bool lmaj = wave < 5 || (wave == 7 && (lane >> 4) < SPL);
    const unsigned esz = lmaj ? E_MAJ : E_MIN;
    unsigned fo[4];
#pragma unroll
    for (int tt = 0; tt < 4; ++tt) {
        const int t = t0 + tt;
        fo[tt] = lane * 16 + (wave < 5 ? (t / 5) * R_MAJ + (t % 5) * 1024
                              : wave < 7 ? 4 * R_MAJ + ((t - 20) >> 1) * R_MIN + ((t - 20) & 1) * 1024
                              : lmaj ? tt * R_MAJ + 5120 : 4 * R_MAJ + tt * R_MIN + 2048 - SPL * 256);
    }
    __amdgpu_buffer_rsrc_t xr = __builtin_amdgcn_make_buffer_rsrc((void*)(me + boff), 0, 5 << 20, 0x00020000);
    const unsigned xs0 = (unsigned)((kind ? 0 : 16) + 4 * rq) * esz;
    UNTIL(tb + NS(T_LANDX) + dq * (unsigned)((kind ? 0 : 16) + 4 * rq + 3));
    while (!dead) {
        unsigned v = tag;
        if ((lane & 12) == 12 && ((lane >> 4) == 3 || (lane >> 4) == SPL - 1)) {
            const int tt = lane & 3;
            v = __builtin_amdgcn_raw_buffer_load_b32(xr, (tt == 0 ? fo[0] : tt == 1 ? fo[1] : tt == 2 ? fo[2] : fo[3]) + xs0 + 3 * esz, 0, 17);
        }
        if (__ballot(((v ^ tag) & TAG16) != 0u) == 0ull) break;
        __builtin_amdgcn_s_sleep(12);
        if (sp.fail()) dead = true;
    }
    u32x4 af[4][4];
    while (!dead) {
        unsigned m = 0u;
#pragma unroll
        for (int tt = 0; tt < 4; ++tt)
#pragma unroll
            for (int a = 0; a < 4; ++a)
                af[tt][a] = __builtin_amdgcn_raw_buffer_load_b128(xr, fo[tt] + xs0 + a * esz, 0, 17);
#pragma unroll
        for (int tt = 0; tt < 4; ++tt)
#pragma unroll
            for (int a = 0; a < 4; ++a) { af[tt][a] ^= tag; m |= or4(af[tt][a]); }
        if (__ballot((m & TAG16) != 0u) == 0ull) break;
        __builtin_amdgcn_s_sleep(12);
        if (sp.fail()) dead = true;
    }
    clean_vm();
    int cx[4][2], ex[2] = {0, 0};
#pragma unroll
    for (int tt = 0; tt < 4; ++tt) {
        cx[tt][0] = code_from(af[tt][0], lane); cx[tt][1] = code_from(af[tt][2], lane);
#pragma unroll
        for (int a = 0; a < 4; ++a) af[tt][a] &= PAYLOAD;
    }
    const char* wq = me + boff;
#pragma unroll
    for (int tt = 0; tt < 4; ++tt) fo[tt] += (unsigned)(32 + c8 * 2 + u0) * esz;
    unsigned due = tb + NS(T_LAND) + (fast ? NS(300) : (kind ? 0u : NS(T_OWN))) + dq * (unsigned)(32 + c8 * 2 + u0);
    unsigned lag = 0;
#pragma unroll 1
    for (int s = 0; s < 8; ++s) {
        const unsigned g = due >> 8;
        due += 16 * dq;
        while (NOW() < g + lag) __builtin_amdgcn_s_sleep(1);
        lag = lag > LAG_DEC / 10 ? lag - LAG_DEC / 10 : 0;
        u32x4 cur[4];
        for (;;) {
            const unsigned t_try = NOW();
            asm volatile("s_nop 4\n\tglobal_load_dwordx4 %0, %4, %8 sc0 sc1\n\tglobal_load_dwordx4 %1, %5, %8 sc0 sc1\n\t"
                         "global_load_dwordx4 %2, %6, %8 sc0 sc1\n\tglobal_load_dwordx4 %3, %7, %8 sc0 sc1\n\t"
                         "s_waitcnt vmcnt(0)"
                         : "=&v"(cur[0]), "=&v"(cur[1]), "=&v"(cur[2]), "=&v"(cur[3])
                         : "v"(fo[0]), "v"(fo[1]), "v"(fo[2]), "v"(fo[3]), "s"(wq) : "memory");
#pragma unroll
            for (int tt = 0; tt < 4; ++tt) cur[tt] ^= tag;
            if (dead || __ballot(((or4(cur[0]) | or4(cur[1]) | or4(cur[2]) | or4(cur[3])) & TAG16) != 0u) == 0ull) break;
            lag = t_try + LAG_INC / 10 - g;
            __builtin_amdgcn_s_sleep(6);
            if (sp.fail()) { dead = true; break; }
        }
#pragma unroll
        for (int tt = 0; tt < 4; ++tt) fo[tt] += 16u * esz;
        if (s == 0) {
            bool bad = false, same = true;
#pragma unroll
            for (int tt = 0; tt < 4; ++tt) {
                const int cw = code_from(cur[tt], lane);
                bad |= cw == 255 || cx[tt][0] == 255 || cx[tt][1] == 255;
                cx[tt][0] += cw; cx[tt][1] += cw;
            }
            ex[0] = __builtin_amdgcn_readfirstlane(cx[0][0]); ex[1] = __builtin_amdgcn_readfirstlane(cx[0][1]);
#pragma unroll
            for (int tt = 0; tt < 4; ++tt) same &= cx[tt][0] == ex[0] && cx[tt][1] == ex[1];
            if (__ballot(bad || !same) != 0ull) {
                const bool poison = __ballot(bad) != 0ull;
#pragma unroll
                for (int hh = 0; hh < 2; ++hh) {
                    ex[hh] = (int)wave_max_u((unsigned)max(max(cx[0][hh], cx[1][hh]), max(cx[2][hh], cx[3][hh])));
#pragma unroll
                    for (int tt = 0; tt < 4; ++tt) {
                        const _Float16 m = poison ? (_Float16)__builtin_nanf("") : (_Float16)__builtin_ldexpf(1.f, cx[tt][hh] - ex[hh]);
#pragma unroll
                        for (int a = 0; a < 2; ++a) {
                            f16x8 t = __builtin_bit_cast(f16x8, af[tt][2 * hh + a]);
                            t *= m;
                            af[tt][2 * hh + a] = __builtin_bit_cast(u32x4, t);
                        }
                    }
                }
            }
            ex[0] -= 252; ex[1] -= 252;
        }
#pragma unroll
        for (int tt = 0; tt < 4; ++tt) cur[tt] &= PAYLOAD;
        f32x4 acc[4];
#pragma unroll
        for (int a = 0; a < 4; ++a) acc[a] = f32x4{0.f, 0.f, 0.f, 0.f};
#pragma unroll
        for (int tt = 0; tt < 4; ++tt)
#pragma unroll
            for (int a = 0; a < 4; ++a)
                acc[a] = __builtin_amdgcn_mfma_f32_16x16x32_f16(__builtin_bit_cast(f16x8, cur[tt]), __builtin_bit_cast(f16x8, af[tt][a]), acc[a], 0, 0, 0);
        if (s >= 4) LDS_WAIT(8 + s - 4, 4);
        asm volatile("" ::: "memory");
        char* rp = smem + L_RED + (s & 3) * 32768 + wave * 4096 + lane * 16;
#pragma unroll
        for (int a = 0; a < 4; ++a) {
            const int e = ex[a >> 1];
            *(f32x4*)(rp + a * 1024) = f32x4{__builtin_ldexpf(acc[a][0], e), __builtin_ldexpf(acc[a][1], e),
                                             __builtin_ldexpf(acc[a][2], e), __builtin_ldexpf(acc[a][3], e)};
        }
        asm volatile("" ::: "memory");
        LDS_SIGNAL(s);
    }
}

}

extern "C" void kernel_launch(int rank, int nranks, ncclComm_t comm,
                              hipStream_t stream,
                              void* const* d_in, const int* in_sizes, int n_in,
                              void* d_out, int out_size,
                              const int* in_dtypes, int out_dtype,
                              void* const* d_out_all,
                              void* const* d_ws_all, size_t ws_bytes) {
    if (nranks != NR || (unsigned)rank >= (unsigned)NR || ws_bytes < WS_NEED || n_in < 2) {
        fprintf(stderr, "unexpected configuration\n");
        return;
    }
    Params p;
    p.x = (const float*)d_in[0];
    p.w = (const float*)d_in[1];
    p.out = (float*)d_out;
    p.me = (char*)d_ws_all[rank];
    for (int g = 0; g < 2; ++g)
        for (int d = 0; d < 4; ++d) p.rel[g][d] = (char*)d_ws_all[(((rank >> 2) ^ g) << 2) | ((rank + 1 + d) & 3)];
    p.rank = rank;
    hipLaunchKernelGGL(fused_kernel, dim3(NWG), dim3(NTHREADS), 0, stream, p);
    HIP_CHECK(hipGetLastError());
}
```
